```python
import numpy as np

kernel_name = "gpu_gemm_a2a_m4096_k4096_n16384_w8a8_dqrelu"


ABSMAX_RTOL = 0.001

COMPUTE_DTYPE = "int8"
EPILOGUE = "dequant_relu"
IN_QUANT = "int8"
OUT_QUANT = None
IN_DTYPES = {"x": "int8", "w_mat": "int8",
             "scale_x": "float32", "scale_w": "float32"}
WORLD_SIZE = 8

def setup_inputs():

    x = np.random.randint(-127, 128, size=(4096, 4096)).astype(np.int8)
    w_mat = np.random.randint(-127, 128, size=(4096, 16384)).astype(np.int8)
    scale_x = (0.02 * (1.0 + np.random.rand(1))).astype(np.float32)
    scale_w = (0.02 * (1.0 + np.random.rand(1))).astype(np.float32)
    return {"x": x, "w_mat": w_mat, "scale_x": scale_x, "scale_w": scale_w}

def reference(x, w_mat, scale_x, scale_w):
    acc = x.astype(np.float64) @ w_mat.astype(np.float64)
    y = (acc.astype(np.float32) * (scale_x * scale_w)).astype(np.float32)
    return np.maximum(y, 0.0).astype(np.float32)
```

```cpp
#define DIAG_PRINT 1
#include <hip/hip_runtime.h>
#include <rccl/rccl.h>
#include <cstdio>
#include <cstdint>
#include <utility>

#define HIP_CHECK(expr) do { hipError_t _e = (expr); if (_e != hipSuccess) { \
    fprintf(stderr, "HIP error %s at %s:%d\n", hipGetErrorString(_e), __FILE__, __LINE__); \
    return; } } while (0)

typedef unsigned u32x4 __attribute__((ext_vector_type(4)));
typedef int i32x4 __attribute__((ext_vector_type(4)));
typedef float f32x4 __attribute__((ext_vector_type(4)));
#define GAS __attribute__((address_space(1)))
#define LAS __attribute__((address_space(3)))
#define DI __device__ __forceinline__
#define SCHED() __builtin_amdgcn_sched_barrier(0)

constexpr int NR = 8;
constexpr int MS = 512, KD = 4096;
constexpr int NFULL = 16384, NS = 2048;
constexpr int NCH = 16;
constexpr int PROG = 4 * NCH;
constexpr unsigned CH_BYTES = 32u * KD;
constexpr unsigned SHARD_BYTES = MS * KD;
constexpr int NWG = 256, NTHR = 256;
constexpr size_t WS_CTR = 0;
constexpr size_t WS_ROLE = 64u << 10;
constexpr size_t WS_FLAGS = 128u << 10;
constexpr unsigned F_TMO = NR * 4 * 128;
constexpr unsigned F_LOC = F_TMO + 128;
constexpr unsigned F_BYTES = F_LOC + 7 * 4 * 128;
constexpr size_t WS_DBG = 256u << 10;
constexpr size_t WS_DATA = 1u << 20;
constexpr size_t WS_NEED = WS_DATA + 2ull * NR * SHARD_BYTES;
constexpr unsigned long long DEADLINE_TICKS = 50000000ull;
constexpr unsigned RED_ROW = 272, RED_WAVE = 32 * RED_ROW, RED_BUF = 4 * RED_WAVE;
constexpr unsigned LDS_CPY = 131072, LDS_BAD = LDS_CPY + 2 * 8192, LDS_BYTES = LDS_BAD + 16;

#define MAP_MODE 0
#define XAUX "sc0 sc1"
#define OUT_AUX 17
#define WPF 1

struct Params {
  const signed char* x;
  char* myws;
  int rank;
  unsigned xcc_tab;
  const signed char* w;
  const float* sx;
  const float* sw;
  float* out;
  char* pad;
  char* wsrel[NR];
};

#define R10(p, x) p #x "0", p #x "1", p #x "2", p #x "3", p #x "4", p #x "5", p #x "6", p #x "7", p #x "8", p #x "9"
#define ACLOB "a0", "a1", "a2", "a3", "a4", "a5", "a6", "a7", "a8", "a9", R10("a", 1), R10("a", 2), R10("a", 3), R10("a", 4), \
    R10("a", 5), R10("a", 6), R10("a", 7), R10("a", 8), R10("a", 9), R10("a", 10), R10("a", 11), R10("a", 12), R10("a", 13), \
    R10("a", 14), R10("a", 15), R10("a", 16), R10("a", 17), R10("a", 18), R10("a", 19), R10("a", 20), R10("a", 21), \
    R10("a", 22), R10("a", 23), R10("a", 24), "a250", "a251", "a252", "a253", "a254", "a255"
#define VCLOB "v192", "v193", "v194", "v195", "v196", "v197", "v198", "v199", R10("v", 20), R10("v", 21), R10("v", 22), \
    R10("v", 23), R10("v", 24), "v250", "v251", "v252", "v253", "v254", "v255"

template <int... Is, typename F>
DI void static_for_impl(std::integer_sequence<int, Is...>, F&& f) { (f(std::integral_constant<int, Is>{}), ...); }
template <int N, typename F>
DI void static_for(F&& f) { static_for_impl(std::make_integer_sequence<int, N>{}, f); }

DI unsigned rfl(unsigned v) { return __builtin_amdgcn_readfirstlane(v); }

#define TS(i) do {} while (0)
#define TC(i) do {} while (0)

template <int N> DI void wait_vm() { asm volatile("s_waitcnt vmcnt(%c0)" :: "i"(N) : "memory"); }

template <int OFF> DI void gstore_sys(const u32x4& r, unsigned voff, char* sbase) {
  asm volatile("global_store_dwordx4 %0, %1, %2 offset:%c3 sc0 sc1\n\ts_nop 1" :: "v"(voff), "v"(r), "s"(sbase), "i"(OFF) : "memory");
}
DI void dma16(unsigned voff, const signed char* sbase, unsigned ldsaddr) {
  asm volatile("s_mov_b32 m0, %2\n\ts_nop 0\n\tglobal_load_lds_dwordx4 %0, %1" :: "v"(voff), "s"(sbase), "s"(ldsaddr) : "memory");
}
DI void flag_store(unsigned vzero, unsigned val, char* sflag) {
  asm volatile("s_mov_b64 exec, 1\n\tglobal_store_dword %0, %1, %2 sc0 sc1\n\ts_mov_b64 exec, -1"
               :: "v"(vzero), "v"(val), "s"(sflag) : "memory");
}

template <int AREG, int OFF> DI void asm_tr8(unsigned addr) {
  asm volatile("ds_read_b64_tr_b8 a[%c0:%c1], %2 offset:%c3" :: "i"(AREG), "i"(AREG + 1), "v"(addr), "i"(OFF) : "memory");
}
template <int F> DI void mfma2(const u32x4& xf) {
  if constexpr (F == 0)
    asm volatile("v_mfma_i32_32x32x32_i8 v[192:207], a[%c1:%c2], %0, 0\n\tv_mfma_i32_32x32x32_i8 v[208:223], a[%c3:%c4], %0, 0"
                 :: "v"(xf), "i"(8 * F), "i"(8 * F + 3), "i"(8 * F + 4), "i"(8 * F + 7));
  else if constexpr (F == 31)
    asm volatile("v_mfma_i32_32x32x32_i8 v[224:239], a[%c1:%c2], %0, v[192:207]\n\tv_mfma_i32_32x32x32_i8 v[240:255], a[%c3:%c4], %0, v[208:223]"
                 :: "v"(xf), "i"(8 * F), "i"(8 * F + 3), "i"(8 * F + 4), "i"(8 * F + 7));
  else
    asm volatile("v_mfma_i32_32x32x32_i8 v[192:207], a[%c1:%c2], %0, v[192:207]\n\tv_mfma_i32_32x32x32_i8 v[208:223], a[%c3:%c4], %0, v[208:223]"
                 :: "v"(xf), "i"(8 * F), "i"(8 * F + 3), "i"(8 * F + 4), "i"(8 * F + 7));
}
template <int J> DI void stage_write(unsigned addr) {
  asm volatile("ds_write_b128 %0, v[%c1:%c2] offset:%c3" :: "v"(addr), "i"(224 + 4 * J), "i"(227 + 4 * J), "i"(32 * J) : "memory");
}
template <int OFF> DI void asm_xload(const u32x4& r, unsigned voff, u32x4 rs, unsigned soff) {
  asm volatile("buffer_load_dwordx4 %0, %1, %2, %3 offen offset:%c4 " XAUX :: "v"(r), "v"(voff), "s"(rs), "s"(soff), "i"(OFF) : "memory");
}
DI void asm_fload(const unsigned& r, unsigned voff, u32x4 rs) {
  asm volatile("buffer_load_dword %0, %1, %2, 0 offen sc0 sc1" :: "v"(r), "v"(voff), "s"(rs) : "memory");
}
template <int N> DI unsigned wait_mov(const unsigned& r) {
  unsigned o;
  asm volatile("s_waitcnt vmcnt(%c2)\n\tv_mov_b32 %0, %1\n\ts_nop 1" : "=v"(o) : "v"(r), "i"(N) : "memory");
  return o;
}
DI u32x4 make_srd(const void* base, unsigned bytes) {
  const unsigned long long a = (unsigned long long)base;
  return u32x4{rfl((unsigned)a), rfl((unsigned)(a >> 32) & 0xffffu), bytes, 0x00020000u};
}
DI void read_unit(u32x4 (&r)[8], unsigned a) {
  asm volatile("ds_read_b128 %0, %8\n\tds_read_b128 %1, %9\n\tds_read_b128 %2, %10\n\tds_read_b128 %3, %11\n\t"
               "ds_read_b128 %4, %12\n\tds_read_b128 %5, %13\n\tds_read_b128 %6, %14\n\tds_read_b128 %7, %15\n\ts_waitcnt lgkmcnt(0)"
               : "=&v"(r[0]), "=&v"(r[1]), "=&v"(r[2]), "=&v"(r[3]), "=&v"(r[4]), "=&v"(r[5]), "=&v"(r[6]), "=&v"(r[7])
               : "v"(a), "v"(a ^ 16), "v"(a ^ 32), "v"(a ^ 48), "v"(a ^ 64), "v"(a ^ 80), "v"(a ^ 96), "v"(a ^ 112) : "memory");
}
DI unsigned unit_src(int j, int lane) {
  const unsigned row = 4 * j + (lane >> 4);
  return row * KD + (((lane & 15) ^ (row & 15)) << 4);
}
DI unsigned unit_rd(int lane) { return (lane & 31) * 256 + ((((lane >> 5) * 8) ^ (lane & 15)) << 4); }

struct Epi {
  i32x4 t[8];
  unsigned bad;
  i32x4 s;
  f32x4 y;
};

__global__ void __launch_bounds__(NTHR, 1) gemm_a2a_kernel(Params p) {
  __shared__ __attribute__((aligned(16))) char lds[LDS_BYTES];
  const int tid = threadIdx.x, lane = tid & 63;
  const int w = rfl(tid >> 6);
  const int b = blockIdx.x, slot = b >> 3;
  const unsigned rank = (unsigned)p.rank & (NR - 1);
  char* myws = p.myws;
  const unsigned lds0 = (unsigned)(uintptr_t)(LAS char*)lds;
  TS(0);
  TC(42);

  if (slot < 4) {
    if (w != 0) return;
    const int kq = slot;
    unsigned xcc;
    asm volatile("s_getreg_b32 %0, hwreg(HW_REG_XCC_ID)" : "=s"(xcc));
    unsigned drel = (p.xcc_tab >> (4 * (xcc & 7))) & 7;
    char* dws = p.wsrel[drel];
    unsigned tick;
    asm volatile("" : "=v"(tick));
    {
      const unsigned off = lane == 0 ? (unsigned)(WS_CTR + b * 256) : (unsigned)(WS_ROLE + (kq * NR + drel) * 64);
      const unsigned one = 1;
      asm volatile("s_mov_b64 exec, 3\n\tglobal_atomic_add %0, %1, %2, %3 sc0\n\ts_mov_b64 exec, -1"
                   :: "v"(tick), "v"(off), "v"(one), "s"(myws) : "memory");
    }
    unsigned vsrc[8];
#pragma unroll
    for (int j = 0; j < 8; ++j) vsrc[j] = unit_src(j, lane);
    const signed char* xq = p.x + kq * 1024;
#pragma unroll
    for (int u = 0; u < 3; ++u)
#pragma unroll
      for (int j = 0; j < 8; ++j) dma16(vsrc[j], xq + u * 256, lds0 + u * 8192 + j * 1024);
    const unsigned vrd = lds0 + unit_rd(lane);
    const unsigned tk = wait_mov<24>(tick);
    const unsigned prev = __builtin_amdgcn_readlane(tk, 0);
    if (__builtin_amdgcn_readlane(tk, 1) != prev) {
      char* roles = myws + WS_ROLE + kq * (NR * 64);
      unsigned got = 0;
      if (lane == 0) {
        __hip_atomic_fetch_sub((GAS unsigned*)(roles + drel * 64), 1u, __ATOMIC_RELAXED, __HIP_MEMORY_SCOPE_AGENT);
        for (unsigned i = 0; i < NR; ++i) {
          unsigned expect = prev;
          if (__hip_atomic_compare_exchange_strong((GAS unsigned*)(roles + i * 64), &expect, prev + 1, __ATOMIC_RELAXED,
                                                   __ATOMIC_RELAXED, __HIP_MEMORY_SCOPE_AGENT)) { got = i; break; }
        }
      }
      drel = rfl(got) & 7;
      dws = p.wsrel[drel];
      wait_vm<0>();
    }
    if (drel == 0) { wait_vm<0>(); TS(40); return; }
    char* dst = dws + WS_DATA + (size_t)(prev & 1) * (NR * SHARD_BYTES) + (size_t)rank * SHARD_BYTES + kq * 32768;
    char* flag = dws + WS_FLAGS + (rank * 4 + kq) * 128;
    const unsigned ldst = lane * 16 + 4096;
    const unsigned vzero = 0;
    const unsigned base = prev * PROG;
    wait_vm<16>();
    for (int u = 0; u < PROG; ++u) {
      const int un = min(u + 3, PROG - 1);
      const signed char* s = xq + ((unsigned)(un >> 2) * CH_BYTES + (un & 3) * 256);
      char* dd = dst + ((unsigned)(u >> 2) * CH_BYTES + (u & 3) * 8192);
      const unsigned lw = lds0 + ((u + 3) & 3) * 8192;
      u32x4 r[8];
      wait_vm<23>();
      read_unit(r, vrd + (u & 3) * 8192);
      static_for<8>([&](auto I) {
        constexpr int i = I;
        if constexpr (i > 0) wait_vm<23>();
        if constexpr (i == 3) {
          if (u >= 2 && (((u - 2) & 3) == 3 || u - 2 >= PROG - 4)) {
            flag_store(vzero, base + u - 1, flag);
          }
        }
        gstore_sys<i * 1024 - 4096>(r[i], ldst, dd);
        dma16(vsrc[i], s, lw + i * 1024);
      });
    }
    wait_vm<17>();
    flag_store(vzero, base + PROG - 1, flag);
    wait_vm<1>();
    flag_store(vzero, base + PROG, flag);
    TS(16);
    wait_vm<0>();
    TS(40);
    return;
  }

  int g, strip;
  {
    const int cj = slot - 4, xcd = b & 7;
    g = cj >> 2; strip = xcd * 4 + (cj & 3);
  }
  const unsigned srcrank = (rank + 1 + (unsigned)g) & (NR - 1);
  float scale = p.sx[0] * p.sw[0];
  LAS unsigned* badword = (LAS unsigned*)((LAS char*)lds + LDS_BAD);
  if (tid == 0) *badword = 0;

  unsigned tick;
  asm volatile("" : "=v"(tick));
  {
    const unsigned off = (unsigned)(WS_CTR + (b * 4 + w) * 64), one = 1;
    asm volatile("s_mov_b64 exec, 1\n\tglobal_atomic_add %0, %1, %2, %3 sc0\n\ts_mov_b64 exec, -1"
                 :: "v"(tick), "v"(off), "v"(one), "s"(myws) : "memory");
  }

  const int nloc = g < 2 ? 3 : 2;
  const int nitems = NCH + nloc;
  const u32x4 xrs = make_srd(myws + WS_DATA, 2 * NR * SHARD_BYTES);
  const u32x4 frs = make_srd(myws + WS_FLAGS, F_BYTES);
  const unsigned xlane = lane * 16;
  u32x4 xf[32];
#pragma unroll
  for (int f = 0; f < 32; ++f) asm volatile("" : "=v"(xf[f]));
  unsigned fv;
  asm volatile("" : "=v"(fv));
  auto load_frag = [&](unsigned off, auto F) { asm_xload<(F & 3) * 1024>(xf[F], xlane, xrs, off + (F >> 2) * 4096); };
  auto load_all = [&](unsigned off) { static_for<32>([&](auto F) { load_frag(off, F); }); };
  const unsigned loff = F_LOC + (g * 4 + w) * 128 + (unsigned)(lane % (4 * nloc)) * 4;

  unsigned prev;
  bool early;
  {
    const int v = w * 32 + strip;
    const bool copier = v < 16 * nloc;
    const unsigned cch = g + 7 * (v >> 4);
    if (copier) {
      const signed char* s = p.x + (size_t)cch * CH_BYTES + (v & 15) * 256;
#pragma unroll
      for (int j = 0; j < 8; ++j) dma16(unit_src(j, lane), s, lds0 + LDS_CPY + w * 8192 + j * 1024);
    }
    const signed char* wq = p.w + (size_t)(w * 1024) * NFULL + rank * NS;
    {
      const signed char* s = wq + (strip & ~3) * 64;
#pragma unroll
      for (int i = 0; i < 3; ++i) {
        const unsigned o = (((slot - 4) * 48 + 16 * i + (lane >> 2)) & 1023) * NFULL + (lane & 3) * 64;
        asm volatile("global_load_dword %0, %1, %2" :: "v"(fv), "v"(o), "s"(s) : "memory");
      }
    }
    const signed char* wsrc = wq + strip * 64;
    const unsigned vw = (lane >> 2) * NFULL + (lane & 3) * 16;
    const unsigned wl = lds0 + w * 32768;
    auto dma = [&](int rd) {
#pragma unroll
      for (int i = 0; i < 16; ++i) dma16(vw, wsrc + (size_t)(rd * 256 + i * 16) * NFULL, wl + (rd & 1) * 16384 + i * 1024);
    };
    const unsigned raddr = wl + ((lane >> 5) * 128 + ((lane & 15) >> 1)) * 64 + ((lane >> 4) & 1) * 16 + (lane & 1) * 8;
    auto wread = [&](auto RD) {
      constexpr int rd = RD;
      static_for<32>([&](auto I) {
        constexpr int f7 = I >> 2, n = (I >> 1) & 1, t = I & 1;
        constexpr int areg = ((rd * 8 + f7) * 2 + n) * 4 + 2 * t;
        constexpr int off = (rd & 1) * 16384 + (f7 * 16 + 8 * t) * 64 + 32 * n;
        asm_tr8<areg, off>(raddr);
      });
      asm volatile("s_waitcnt lgkmcnt(0)" ::: "memory");
    };
    asm volatile("" ::: ACLOB, VCLOB);
    dma(0);
    prev = rfl(wait_mov<16>(tick));
    if (copier) {
      u32x4 r[8];
      read_unit(r, lds0 + LDS_CPY + w * 8192 + unit_rd(lane));
      char* dd = myws + WS_DATA + (size_t)(prev & 1) * (NR * SHARD_BYTES) + (size_t)rank * SHARD_BYTES + (size_t)cch * CH_BYTES + (v & 15) * 8192;
      static_for<8>([&](auto I) { gstore_sys<I * 1024 - 4096>(r[I], xlane + 4096, dd); });
    }
    dma(1);
    wait_vm<16>();
    if (copier) flag_store(0u, prev + 1, myws + WS_FLAGS + F_LOC + (g * 4 + ((v >> 2) & 3)) * 128 + ((v >> 4) * 4 + (v & 3)) * 4);
    wread(std::integral_constant<int, 0>{});
    dma(2);
    wait_vm<16>();
    wread(std::integral_constant<int, 1>{});
    asm_fload(fv, loff, frs);
    dma(3);
    early = __all((int)(wait_mov<16>(fv) - (prev + 1)) >= 0);
    if (early) load_all((prev & 1) * (NR * SHARD_BYTES) + w * 32768 + (rank * MS + g * 32) * KD);
    wread(std::integral_constant<int, 2>{});
    if (early) wait_vm<32>(); else wait_vm<0>();
    wread(std::integral_constant<int, 3>{});
  }
  const unsigned call = prev + 1;
  asm volatile("" : "+v"(scale));
  TS(1);
  asm volatile("s_waitcnt lgkmcnt(0)\n\ts_barrier" ::: "memory");

  const __amdgpu_buffer_rsrc_t ors = __builtin_amdgcn_make_buffer_rsrc(p.out, 0, (unsigned)(NR * MS) * NS * 4u, 0x00020000);
  GAS unsigned* tmo = (GAS unsigned*)(myws + WS_FLAGS + F_TMO);
  const unsigned base = prev * PROG;
  const unsigned foff = (lane & 16) ? F_TMO : (srcrank * 4 + w) * 128;
  const unsigned pbase = (prev & 1) * (NR * SHARD_BYTES) + w * 32768;

  int known = 0;
  unsigned tmov = 0;
  bool dead = false;
#define FLAG_LOAD() asm_fload(fv, foff, frs)
#define FLAG_TAKE(n) do { const unsigned v_ = wait_mov<n>(fv); known = (int)(__builtin_amdgcn_readlane(v_, 0) - base); \
    tmov = __builtin_amdgcn_readlane(v_, 16); } while (0)
#define SPIN(poll, cond) do { if (!dead) { \
    const unsigned long long t0 = __builtin_amdgcn_s_memrealtime(); \
    for (;;) { \
      poll; \
      if (cond) break; \
      if (tmov == call || __builtin_amdgcn_s_memrealtime() - t0 > DEADLINE_TICKS) { \
        if (lane == 0) { \
          __hip_atomic_store(tmo, call, __ATOMIC_RELAXED, __HIP_MEMORY_SCOPE_AGENT); \
          *badword = 1; \
        } \
        dead = true; \
        break; \
      } \
      __builtin_amdgcn_s_sleep(1); \
    } \
  } } while (0)
#define WAIT_FOR(cond) SPIN(FLAG_LOAD(); FLAG_TAKE(0), cond)

  const unsigned wr_base = lds0 + w * RED_WAVE + (lane & 31) * RED_ROW + (lane >> 5) * 16;
  const unsigned rd_base = (w * 8 + (lane >> 4)) * RED_ROW + (lane & 15) * 16;
  const unsigned ost = (w * 8 + (lane >> 4)) * (NS * 4) + strip * 256 + (lane & 15) * 16;
  Epi e;
  auto epi = [&](auto SL, unsigned rb, unsigned orow) {
    constexpr int s = SL;
    if constexpr (s < 8) stage_write<s>(wr_base + rb);
    else if constexpr (s == 8) asm volatile("s_waitcnt lgkmcnt(0)\n\ts_barrier" ::: "memory");
    else if constexpr (s < 13) {
      constexpr int q = s - 9;
      LAS char* a = (LAS char*)lds + rb + rd_base;
      e.t[2 * q] = *(LAS i32x4*)(a + ((2 * q) & 3) * RED_WAVE + (q >> 1) * 4 * RED_ROW);
      e.t[2 * q + 1] = *(LAS i32x4*)(a + ((2 * q + 1) & 3) * RED_WAVE + (q >> 1) * 4 * RED_ROW);
      if constexpr (q == 0) e.bad = *badword;
    } else {
      constexpr int j = (s - 13) / 3, k = (s - 13) % 3;
      if constexpr (k == 0) {
        e.s = (e.t[4 * j] + e.t[4 * j + 1]) + (e.t[4 * j + 2] + e.t[4 * j + 3]);
        asm volatile("" : "+v"(e.s));
      } else if constexpr (k == 1) {
        const float sc = e.bad ? __builtin_nanf("") : scale;
#pragma unroll
        for (int i = 0; i < 4; ++i) e.y[i] = (float)e.s[i] * sc;
        asm volatile("" : "+v"(e.y));
      } else {
#pragma unroll
        for (int i = 0; i < 4; ++i) e.y[i] = __builtin_elementwise_maximum(e.y[i], 0.f);
        __builtin_amdgcn_raw_buffer_store_b128(__builtin_bit_cast(u32x4, e.y), ors, ost + orow + j * 4 * (NS * 4), 0, OUT_AUX);
      }
    }
  };
  constexpr int NSL = 19;
  auto epi_now = [&](unsigned rb, unsigned orow) {
    asm volatile("s_nop 7\n\ts_nop 7" ::: "memory");
    static_for<NSL>([&](auto S) { epi(S, rb, orow); });
  };
  auto block = [&](auto REFILL, auto PEND, unsigned noff, unsigned rb, unsigned orow) {
    static_for<32>([&](auto F) {
      constexpr int f = F;
      wait_vm<REFILL ? 32 : 32 - f>();
      mfma2<f>(xf[f]);
      if constexpr (REFILL) load_frag(noff, F);
      if constexpr (PEND) {
        if constexpr (f >= 1 && f <= 8) epi(std::integral_constant<int, f - 1>{}, rb, orow);
        else if constexpr (f >= 10 && f <= 14) epi(std::integral_constant<int, f - 2>{}, rb, orow);
        else if constexpr (f >= 17 && f <= 22) epi(std::integral_constant<int, f - 4>{}, rb, orow);
      }
      SCHED();
    });
  };
  using T_ = std::true_type;
  using F_ = std::false_type;

  int nneed; unsigned nrow;
#define NEXT_ITEM(i) do { const int c_ = (i) - nloc; nneed = 4 * c_ + 4; \
    nrow = c_ < 0 ? rank * MS + (unsigned)(g + 7 * (i)) * 32 : srcrank * MS + (unsigned)c_ * 32; } while (0)
#define NREADY() (known >= nneed)

  NEXT_ITEM(0);
  if (!early) {
    bool ok;
    SPIN(asm_fload(fv, loff, frs); ok = __all((int)(wait_mov<0>(fv) - call) >= 0); FLAG_LOAD(); FLAG_TAKE(0), ok);
    load_all(pbase + nrow * KD);
  }
  TS(2);
  TC(43);
  bool pend = false;
  unsigned prow = 0;
  for (int it = 0;; ++it) {
    const unsigned crow = nrow * (NS * 4);
    const bool more = it + 1 < nitems;
    NEXT_ITEM(it + 1);
    const unsigned noff = pbase + nrow * KD;
    const unsigned rb = (it & 1) * RED_BUF;
    FLAG_LOAD();
    if (more && (dead || NREADY())) {
      if (pend) block(T_{}, T_{}, noff, RED_BUF - rb, prow); else block(T_{}, F_{}, noff, 0u, 0u);
      FLAG_TAKE(32);
      pend = true;
      prow = crow;
      TS(3 + it);
      continue;
    }
    if (pend) block(F_{}, T_{}, 0u, RED_BUF - rb, prow); else block(F_{}, F_{}, 0u, 0u, 0u);
    pend = false;
    FLAG_TAKE(0);
    const bool rdy = more && NREADY();
    if (rdy) load_all(noff);
    epi_now(rb, crow);
    TS(3 + it);
    if (!more) break;
    if (rdy) continue;
    if (it + 2 < nitems) {
      WAIT_FOR(NREADY());
      load_all(noff);
      continue;
    }
    static_for<4>([&](auto Q) {
      constexpr int q = Q;
      if (known < PROG - 3 + q) WAIT_FOR(known >= PROG - 3 + q);
      static_for<8>([&](auto I) { load_frag(noff, std::integral_constant<int, 8 * q + I>{}); });
      if constexpr (q > 0) {
        static_for<8>([&](auto I) { wait_vm<15 - I>(); mfma2<8 * (q - 1) + I>(xf[8 * (q - 1) + I]); });
      }
    });
    static_for<8>([&](auto I) { wait_vm<7 - I>(); mfma2<24 + I>(xf[24 + I]); });
    epi_now(RED_BUF - rb, nrow * (NS * 4));
    TS(4 + it);
    break;
  }
  TS(40);
  TC(44);
}

extern "C" void kernel_launch(int rank, int nranks, ncclComm_t comm,
                              hipStream_t stream,
                              void* const* d_in, const int* in_sizes, int n_in,
                              void* d_out, int out_size,
                              const int* in_dtypes, int out_dtype,
                              void* const* d_out_all,
                              void* const* d_ws_all, size_t ws_bytes) {
  (void)comm; (void)in_dtypes; (void)out_dtype; (void)d_out_all;
  if (nranks != NR || n_in != 4 || (unsigned)rank >= (unsigned)NR || ws_bytes < WS_NEED ||
      in_sizes[0] != MS * KD || in_sizes[1] != KD * NFULL || out_size != NR * MS * NS) {
    fprintf(stderr, "kernel_launch: unexpected problem shape\n");
    return;
  }
  {
    int dev = -1, cnt = -1; char id[64] = {0};
    (void)hipGetDevice(&dev); (void)hipGetDeviceCount(&cnt); (void)hipDeviceGetPCIBusId(id, 64, dev);
    fprintf(stderr, "DIAG rank %d dev %d of %d pci %s ws_bytes %zu x %p w %p out %p ws %p\n", rank, dev, cnt, id, ws_bytes, d_in[0], d_in[1], d_out, d_ws_all[rank]);
    fprintf(stdout, "DIAG rank %d dev %d of %d pci %s\n", rank, dev, cnt, id);
  }
  Params p;
  p.x = (const signed char*)d_in[0];
  p.w = (const signed char*)d_in[1];
  p.sx = (const float*)d_in[2];
  p.sw = (const float*)d_in[3];
  p.out = (float*)d_out;
  for (int i = 0; i < NR; ++i) p.wsrel[i] = (char*)d_ws_all[(rank + i) % NR];
  p.myws = p.wsrel[0];
  p.pad = nullptr;
  p.rank = rank;
  {
    const unsigned set1 = 0x4D;
    unsigned tab = 0;
    int na = 0, nb = 4;
    for (int dr = 1; dr < NR; ++dr) {
      const int d = (rank + dr) % NR;
      const bool same = ((set1 >> d) & 1) == ((set1 >> rank) & 1);
      tab |= (unsigned)dr << (4 * (same ? na++ : nb++));
    }
    p.xcc_tab = tab;
  }
  hipLaunchKernelGGL(gemm_a2a_kernel, dim3(NWG), dim3(NTHR), 0, stream, p);
  HIP_CHECK(hipGetLastError());
}
```

```cpp
#include <hip/hip_runtime.h>
#include <rccl/rccl.h>
#include <cstdio>
#include <cstddef>
#include <cstdint>
#include <utility>

#define HIP_CHECK(expr) do { hipError_t _e = (expr); if (_e != hipSuccess) { \
    fprintf(stderr, "HIP error %s at %s:%d\n", hipGetErrorString(_e), __FILE__, __LINE__); \
    return; } } while (0)

typedef unsigned u32x4 __attribute__((ext_vector_type(4)));
#define GAS __attribute__((address_space(1)))
#define LAS __attribute__((address_space(3)))
#define DI __device__ __forceinline__
#define SCHED() __builtin_amdgcn_sched_barrier(0)

constexpr int NR = 8;
constexpr int MS = 512, KD = 4096;
constexpr int NFULL = 16384, NS = 2048;
constexpr int NCH = 16;
constexpr int PROG = 4 * NCH;
constexpr unsigned CH_BYTES = 32u * KD;
constexpr unsigned SHARD_BYTES = MS * KD;
constexpr int NWG = 256, NTHR = 256;
constexpr size_t WS_CTR = 0;
constexpr size_t WS_ROLE = 128u << 10;
constexpr size_t WS_FLAGS = 512u << 10;
constexpr unsigned FSTR = 512;
constexpr unsigned F_TMO = NR * 4 * FSTR;
constexpr unsigned F_LOC = F_TMO + 256;
constexpr unsigned F_BYTES = F_LOC + 256 * 128;
constexpr size_t WS_DATA = 1u << 20;
constexpr unsigned KQ_BYTES = SHARD_BYTES / 4, H_BYTES = KQ_BYTES / 2;
constexpr unsigned X_BYTES = 2u * NR * SHARD_BYTES;
constexpr size_t WS_NEED = WS_DATA + X_BYTES;
constexpr unsigned long long DEADLINE_TICKS = 50000000ull;
constexpr unsigned LDS_RED = 131072, LDS_BYTES = 163840;

constexpr int FLAG_GR = 4, FINE_FROM = 44;
constexpr int EARLY_FROM = 54, EARLY_BY = 2;
constexpr int WIN0 = 10, WIN1 = 12, WIN2 = 16, U1 = 48;
constexpr int WLAG = 2;
static_assert(WIN0 > 8 && WIN2 <= 16, "");
constexpr int CLR0 = 6, CLR1 = CLR0 + 2 * (PROG - EARLY_FROM);
static_assert(CLR1 <= FINE_FROM, "");
constexpr unsigned PACE_GOAL = 3650;

struct Params {
  const signed char* x;
  char* myws;
  int rank;
  unsigned xcc_tab;
  const signed char* w;
  const float* sx;
  const float* sw;
  float* out;
  char* pad;
  char* wsx[NR];
};

static_assert(offsetof(Params, myws) == 8 && offsetof(Params, rank) == 16 && offsetof(Params, xcc_tab) == 20 &&
              offsetof(Params, wsx) == 64, "");

#define R10(p, x) p #x "0", p #x "1", p #x "2", p #x "3", p #x "4", p #x "5", p #x "6", p #x "7", p #x "8", p #x "9"
#define ACLOB "a0", "a1", "a2", "a3", "a4", "a5", "a6", "a7", "a8", "a9", R10("a", 1), R10("a", 2), R10("a", 3), R10("a", 4), \
    R10("a", 5), R10("a", 6), R10("a", 7), R10("a", 8), R10("a", 9), R10("a", 10), R10("a", 11), R10("a", 12), R10("a", 13), \
    R10("a", 14), R10("a", 15), R10("a", 16), R10("a", 17), R10("a", 18), R10("a", 19), R10("a", 20), R10("a", 21), \
    R10("a", 22), R10("a", 23), R10("a", 24), "a250", "a251", "a252", "a253", "a254", "a255"
#define VCLOB R10("v", 1), R10("v", 2), R10("v", 3), R10("v", 4), R10("v", 5), R10("v", 6), R10("v", 7), \
    R10("v", 8), R10("v", 9), R10("v", 10), R10("v", 11), R10("v", 12), R10("v", 13), R10("v", 14), R10("v", 15), \
    R10("v", 16), R10("v", 17), R10("v", 18), R10("v", 19), R10("v", 20), R10("v", 21), R10("v", 22), R10("v", 23), \
    R10("v", 24), "v250", "v251", "v252", "v253", "v254", "v255"
#define SCLOB "s64", "s65", "s66", "s67", "s68", "s69", R10("s", 7), R10("s", 8), R10("s", 9)

template <int... Is, typename F>
DI void static_for_impl(std::integer_sequence<int, Is...>, F&& f) { (f(std::integral_constant<int, Is>{}), ...); }
template <int N, typename F>
DI void static_for(F&& f) { static_for_impl(std::make_integer_sequence<int, N>{}, f); }

DI unsigned rfl(unsigned v) { return __builtin_amdgcn_readfirstlane(v); }
DI int lane_id() {
  int l;
  asm volatile("v_mbcnt_lo_u32_b32 %0, -1, 0\n\tv_mbcnt_hi_u32_b32 %0, -1, %0" : "=v"(l));
  return l;
}

template <int N> DI void wait_vm() { asm volatile("s_waitcnt vmcnt(%c0)" :: "i"(N) : "memory"); }

template <int OFF> DI void gstore_sys(const u32x4& r, unsigned voff, char* sbase) {
  asm volatile("global_store_dwordx4 %0, %1, %2 offset:%c3 sc0 sc1\n\ts_nop 1" :: "v"(voff), "v"(r), "s"(sbase), "i"(OFF) : "memory");
}
DI void dma16(unsigned voff, const signed char* sbase, unsigned ldsaddr) {
  asm volatile("s_mov_b32 m0, %2\n\ts_nop 0\n\tglobal_load_lds_dwordx4 %0, %1" :: "v"(voff), "s"(sbase), "s"(ldsaddr) : "memory");
}
DI void unit_src(unsigned (&v)[4], int lane) {
  const unsigned v0 = (lane >> 4) * KD + (((lane & 15) ^ (lane >> 4)) << 4);
#pragma unroll
  for (int j = 0; j < 4; ++j) v[j] = (v0 ^ (j << 6)) + j * (4 * KD - 1024);
}
DI void dma_unit(const unsigned (&v)[4], const signed char* sbase, unsigned ldsaddr) {
  asm volatile("s_mov_b32 m0, %6\n\ts_nop 0\n\tglobal_load_lds_dwordx4 %0, %4\n\tglobal_load_lds_dwordx4 %1, %4 offset:1024\n\t"
               "global_load_lds_dwordx4 %2, %4 offset:2048\n\tglobal_load_lds_dwordx4 %3, %4 offset:3072\n\t"
               "s_add_u32 m0, %6, 4096\n\ts_nop 0\n\tglobal_load_lds_dwordx4 %0, %5\n\tglobal_load_lds_dwordx4 %1, %5 offset:1024\n\t"
               "global_load_lds_dwordx4 %2, %5 offset:2048\n\tglobal_load_lds_dwordx4 %3, %5 offset:3072"
               :: "v"(v[0]), "v"(v[1]), "v"(v[2]), "v"(v[3]), "s"(sbase), "s"(sbase + 16 * KD), "s"(ldsaddr) : "memory", "scc");
}
DI void flag_store(unsigned vzero, unsigned val, char* sflag) {
  asm volatile("s_mov_b64 exec, 1\n\tglobal_store_dword %0, %1, %2 sc0 sc1\n\ts_mov_b64 exec, -1"
               :: "v"(vzero), "v"(val), "s"(sflag) : "memory");
}

template <int N> DI unsigned wait_mov(const unsigned& r) {
  unsigned o;
  asm volatile("s_waitcnt vmcnt(%c2)\n\tv_mov_b32 %0, %1\n\ts_nop 1" : "=v"(o) : "v"(r), "i"(N) : "memory");
  return o;
}
DI u32x4 make_srd(const void* base, unsigned bytes) {
  const unsigned long long a = (unsigned long long)base;
  return u32x4{rfl((unsigned)a), rfl((unsigned)(a >> 32) & 0xffffu), bytes, 0x00020000u};
}
DI void read_unit(u32x4 (&r)[8], unsigned a) {
  asm volatile("ds_read_b128 %0, %8\n\tds_read_b128 %1, %8 offset:4096\n\tds_read_b128 %2, %9\n\tds_read_b128 %3, %9 offset:4096\n\t"
               "ds_read_b128 %4, %10\n\tds_read_b128 %5, %10 offset:4096\n\tds_read_b128 %6, %11\n\tds_read_b128 %7, %11 offset:4096\n\t"
               "s_waitcnt lgkmcnt(0)"
               : "=&v"(r[0]), "=&v"(r[1]), "=&v"(r[2]), "=&v"(r[3]), "=&v"(r[4]), "=&v"(r[5]), "=&v"(r[6]), "=&v"(r[7])
               : "v"(a), "v"(a ^ 64), "v"(a ^ 128), "v"(a ^ 192) : "memory");
}
template <int OFF> DI void lds_frag(const u32x4& r, unsigned a) {
  asm volatile("ds_read_b128 %0, %1 offset:%c2" :: "v"(r), "v"(a), "i"(OFF) : "memory");
}
DI void lds_word(const unsigned& r, unsigned a) { asm volatile("ds_read_b32 %0, %1" :: "v"(r), "v"(a) : "memory"); }
template <int N> DI void wait_vm_l7() { asm volatile("s_waitcnt vmcnt(%c0) lgkmcnt(7)" :: "i"(N) : "memory"); }
template <int N> DI void wait_lgkm() { asm volatile("s_waitcnt lgkmcnt(%c0)" :: "i"(N) : "memory"); }
DI unsigned unit_rd(int lane) { return (lane & 15) * 256 + (((lane >> 4) ^ (lane & 15)) << 4); }

__global__ void __launch_bounds__(NTHR, 1) gemm_a2a_kernel(Params p) {
  __shared__ __attribute__((aligned(128))) char lds[LDS_BYTES];
  unsigned role;
  asm volatile("s_getreg_b32 %0, hwreg(HW_REG_XCC_ID, 0, 3)\n\ts_load_dwordx4 s[88:91], %1, 0x0\n\ts_lshl_b32 s94, %0, 3\n\t"
               "s_load_dwordx2 s[92:93], %1, 0x10\n\ts_load_dwordx2 s[94:95], %1, s94 offset:0x40"
               : "=&s"(role) : "s"(__builtin_amdgcn_kernarg_segment_ptr())
               : "scc", "s88", "s89", "s90", "s91", "s92", "s93", "s94", "s95");
  SCHED();
  const int w = rfl(threadIdx.x >> 6);
  const int b = blockIdx.x, slot = b >> 3;
  const unsigned lds0 = (unsigned)(uintptr_t)(LAS char*)lds;

  if (__builtin_expect(slot < 4, 1)) {
    const int lane = lane_id();
    const int kq = slot;
    {
      unsigned long long pc;
      unsigned t;
      asm volatile("s_cmp_lt_u32 %3, 3\n\ts_cbranch_scc1 1f\n\ts_getpc_b64 %0\n\t"
                   "v_min_i32 %1, gemm_a2a_code_end-128-., %2\n\tv_max_i32 %1, 0, %1\n\tglobal_load_dword v255, %1, %0\n\t"
                   "global_load_dword v255, %1, %0 offset:128\n\ts_endpgm\n1:"
                   : "=&s"(pc), "=&v"(t) : "v"((slot * 64 + lane) * 256), "s"(w) : "memory", "scc");
    }
    unsigned vsrc[4];
    unit_src(vsrc, lane);
    const unsigned ctr = (unsigned)(WS_CTR + b * 512), rol = (unsigned)(WS_ROLE + (kq * NR + role) * 128);
    const unsigned aoff = ctr + (rol - ctr) * lane, one = 1;
    const unsigned toff = (unsigned)WS_DATA + lane * (NR * SHARD_BYTES) + kq * KQ_BYTES + H_BYTES - 128;
    asm volatile("" :: "v"(vsrc[0]), "v"(vsrc[1]), "v"(vsrc[2]), "v"(vsrc[3]), "v"(aoff), "v"(one), "v"(toff));
    char *dws, *myws;
    const signed char* xq;
    unsigned rank, tab;
    asm volatile("s_waitcnt lgkmcnt(0)\n\ts_add_u32 s88, s88, %5\n\ts_addc_u32 s89, s89, 0\n\ts_mov_b64 %0, s[88:89]\n\t"
                 "s_mov_b64 %1, s[90:91]\n\ts_and_b32 %2, s92, 7\n\ts_mov_b32 %3, s93\n\ts_mov_b64 %4, s[94:95]"
                 : "=&s"(xq), "=&s"(myws), "=&s"(rank), "=&s"(tab), "=&s"(dws) : "s"(kq * 1024) : "scc", "s88", "s89");
    unsigned tick, keep;
    asm volatile("" : "=v"(tick), "=v"(keep));
    asm volatile("s_cmp_lg_u32 %0, 0\n\ts_cbranch_scc1 1f\n\ts_mov_b64 exec, 3\n\tglobal_atomic_add %1, %2, %3, %4 sc0\n\t"
                 "s_mov_b64 exec, -1\n\ts_mov_b32 m0, %11\n\ts_nop 0\n\tglobal_load_lds_dwordx4 %5, %9\n\t"
                 "global_load_lds_dwordx4 %6, %9 offset:1024\n\tglobal_load_lds_dwordx4 %7, %9 offset:2048\n\t"
                 "global_load_lds_dwordx4 %8, %9 offset:3072\n\ts_add_u32 m0, %11, 4096\n\ts_nop 0\n\t"
                 "global_load_lds_dwordx4 %5, %10\n\tglobal_load_lds_dwordx4 %6, %10 offset:1024\n\t"
                 "global_load_lds_dwordx4 %7, %10 offset:2048\n\tglobal_load_lds_dwordx4 %8, %10 offset:3072\n\t"
                 "s_mov_b64 exec, 3\n\tglobal_load_dword %12, %13, %14 sc0 sc1\n\ts_mov_b64 exec, -1\n1:"
                 :: "s"(w), "v"(tick), "v"(aoff), "v"(one), "s"(myws), "v"(vsrc[0]), "v"(vsrc[1]), "v"(vsrc[2]), "v"(vsrc[3]),
                    "s"(xq), "s"(xq + 16 * KD), "s"(lds0), "v"(keep), "v"(toff), "s"(dws + rank * SHARD_BYTES) : "memory", "scc");
    LAS unsigned* info = (LAS unsigned*)((LAS char*)lds + LDS_RED);
    auto stage = [&](int u) {
      const signed char* s = xq + ((unsigned)(u >> 2) * CH_BYTES + (u & 3) * 256);
      dma_unit(vsrc, s, lds0 + (u & 15) * 8192);
    };
    if (w != 0) {
      if (w == 1) {
        __builtin_amdgcn_s_sleep(3);
#pragma unroll 1
        for (int u = 1; u < 8; ++u) stage(u);
        wait_vm<40>();
      }
      asm volatile("s_barrier" ::: "memory");
      role = rfl(info[0]) & 7;
      const unsigned prev = rfl(info[1]);
      if (((tab >> (4 * role)) & 7) == 0) { wait_vm<0>(); return; }
      if (w == 1) {
#pragma unroll 1
        for (int u = 1; u + 7 < PROG; ++u) {
          stage(u + 7);
          asm volatile("s_waitcnt vmcnt(40)\n\ts_barrier" ::: "memory");
        }
        asm volatile("s_waitcnt vmcnt(0)\n\ts_barrier" ::: "memory");
        return;
      }
      char* dfl = p.wsx[role] + WS_FLAGS + (rank * 4 + kq) * FSTR;
      char* clr = myws + WS_DATA + (size_t)(~prev & 1) * (NR * SHARD_BYTES) +
                  (size_t)((rank + (tab >> (4 * role))) & (NR - 1)) * SHARD_BYTES + kq * KQ_BYTES + EARLY_FROM * 4096;
      const u32x4 zero = {0, 0, 0, 0};
      unsigned line = 0;
      unsigned c0 = (unsigned)__builtin_amdgcn_s_memtime(), r0 = (unsigned)__builtin_amdgcn_s_memrealtime();
      int hold = 0;
      auto beat = [&](int u, bool flag) {
        asm volatile("s_barrier" ::: "memory");
        if (flag) {
          flag_store(line, prev * PROG + u - WLAG + 1, dfl);
          line = (line + 128) & (FSTR - 1);
        }
        const unsigned c1 = (unsigned)__builtin_amdgcn_s_memtime(), r1 = (unsigned)__builtin_amdgcn_s_memrealtime();
        hold = c1 - c0 < 14 * (r1 - r0) ? 6 : max(hold - 1, 0);
        c0 = c1; r0 = r1;
        info[2] = hold ? 2 : u + 2 < U1 ? 1 : 0;
      };
#pragma unroll 1
      for (int u = 1; u < FINE_FROM + WLAG; ++u) {
        if (u >= CLR0 && u < CLR1)
          static_for<4>([&](auto I) { gstore_sys<(I >> 1) * 1024>(zero, lane * 16 + (I & 1) * H_BYTES, clr + (u - CLR0) * 2048); });
        if (u == FINE_FROM) wait_vm<0>();
        beat(u, u >= WLAG && (u - WLAG + 1) % FLAG_GR == 0);
      }
#pragma unroll 1
      for (int u = FINE_FROM + WLAG; u < PROG + WLAG; ++u) beat(u, true);
      wait_vm<0>();
      return;
    }
    const unsigned tk = wait_mov<9>(tick);
    const unsigned prev = __builtin_amdgcn_readlane(tk, 0);
    if (__builtin_expect(__builtin_amdgcn_readlane(tk, 1) != prev, 0)) {
      char* roles = myws + WS_ROLE + kq * (NR * 128);
      unsigned got = 0;
      if (lane == 0) {
        __hip_atomic_fetch_sub((GAS unsigned*)(roles + role * 128), 1u, __ATOMIC_RELAXED, __HIP_MEMORY_SCOPE_AGENT);
        for (unsigned i = 0; i < NR; ++i) {
          unsigned expect = prev;
          if (__hip_atomic_compare_exchange_strong((GAS unsigned*)(roles + i * 128), &expect, prev + 1, __ATOMIC_RELAXED,
                                                   __ATOMIC_RELAXED, __HIP_MEMORY_SCOPE_AGENT)) { got = i; break; }
        }
      }
      role = rfl(got) & 7;
      dws = p.wsx[role];
      wait_vm<0>();
    }
    if (lane == 0) { info[0] = role; info[1] = prev; info[2] = 1; }
    if (((tab >> (4 * role)) & 7) == 0) {
      asm volatile("s_waitcnt vmcnt(0) lgkmcnt(0)\n\ts_barrier" :: "v"(keep) : "memory");
      return;
    }
    char* dst = dws + WS_DATA + (size_t)(prev & 1) * (NR * SHARD_BYTES) + (size_t)rank * SHARD_BYTES + kq * KQ_BYTES;
    const unsigned ldst[2] = {(unsigned)lane * 16, (unsigned)lane * 16 + H_BYTES};
    const unsigned vrd = lds0 + unit_rd(lane);
    asm volatile("s_waitcnt vmcnt(1)" :: "s"(dst), "v"(ldst[0]), "v"(ldst[1]), "v"(vrd ^ 64), "v"(vrd ^ 128), "v"(vrd ^ 192) : "memory");
    u32x4 r[8];
    unsigned mv;
#pragma unroll
    for (int i = 0; i < 8; ++i) asm volatile("" : "=v"(r[i]));
    asm volatile("" : "=v"(mv));
    auto send = [&](int u, auto W) {
      char* dd = dst + (unsigned)u * 4096;
      const unsigned a = vrd + ((u + 1) & 15) * 8192;
      static_for<8>([&](auto I) {
        constexpr int i = I;
        wait_vm_l7<W - 1>();
        if constexpr (i == (W - 1) % 8) asm volatile("s_barrier" ::: "memory");
        gstore_sys<(i >> 1) * 1024>(r[i], ldst[i & 1], dd);
        lds_frag<(i & 1) * 4096>(r[i], a ^ ((i >> 1) * 64));
        if constexpr (i == 5) lds_word(mv, lds0 + LDS_RED + 8);
      });
    };
    static_for<8>([&](auto I) { lds_frag<(I & 1) * 4096>(r[I], vrd ^ ((I >> 1) * 64)); });
    static_for<8>([&](auto I) {
      wait_lgkm<7 - I>();
      gstore_sys<(I >> 1) * 1024>(r[I], ldst[I & 1], dst);
    });
    asm volatile("s_barrier" ::: "memory");
    static_for<8>([&](auto I) { lds_frag<(I & 1) * 4096>(r[I], (vrd + 8192) ^ ((I >> 1) * 64)); });
    unsigned m = 1;
#pragma unroll 1
    for (int u = 1; u < PROG; ++u) {
      if (m == 1) send(u, std::integral_constant<int, WIN1>{});
      else if (m == 0) send(u, std::integral_constant<int, WIN0>{});
      else send(u, std::integral_constant<int, WIN2>{});
      asm volatile("s_waitcnt lgkmcnt(2)\n\tv_readfirstlane_b32 %0, %1" : "=s"(m) : "v"(mv) : "memory");
    }
    static_for<WLAG>([&](auto K) {
      wait_vm<8 * (WLAG - 1 - K)>();
      asm volatile("s_barrier" ::: "memory");
    });
    asm volatile("s_waitcnt lgkmcnt(0)" :: "v"(keep), "v"(r[6]), "v"(r[7]));
    return;
  }

  const unsigned tgoal = (unsigned)__builtin_amdgcn_s_memrealtime() + PACE_GOAL;
  const int lane = lane_id();
  const unsigned rank = (unsigned)p.rank & (NR - 1);
  char* myws = p.myws;
  int g, h, strip;
  {
    const int cj = slot - 4, xcd = b & 7;
    g = cj >> 2; h = xcd & 1; strip = (xcd >> 1) * 4 + (cj & 3);
  }
  const unsigned srcrank = (rank + 1 + (unsigned)g) & (NR - 1);
  typedef const __attribute__((address_space(4))) float* cfp;
  const float scale = *(cfp)p.sx * *(cfp)p.sw;
  const bool extra = h == 1 && g < 4;
  const int nloc = extra ? 3 : 2;
  const unsigned xlane = lane * 16;
  const unsigned lw = lds0 + w * 32768, lr = lds0 + LDS_RED + w * 8192;
  unsigned tick;
  asm volatile("" : "=v"(tick));
  auto draw = [&]() {
    const unsigned off = (unsigned)(WS_CTR + (b * 4 + w) * 128), one = 1;
    asm volatile("s_mov_b64 exec, 1\n\tglobal_atomic_add %0, %1, %2, %3 sc0\n\ts_mov_b64 exec, -1"
                 :: "v"(tick), "v"(off), "v"(one), "s"(myws) : "memory");
  };
  const unsigned v = extra ? 256 : w * 160 + (h == 1 ? g - 4 : g + 3) * 16 + strip;
  if (v < 256) {
    const unsigned cch = v >> 4;
    draw();
    const signed char* s = p.x + (size_t)cch * CH_BYTES + (v & 15) * 256;
    unsigned vsrc[4];
    unit_src(vsrc, lane);
    dma_unit(vsrc, s, lw);
    const unsigned pv = rfl(wait_mov<0>(tick));
    u32x4 r[8];
    read_unit(r, lw + unit_rd(lane));
    char* dd = myws + WS_DATA + (size_t)(pv & 1) * (NR * SHARD_BYTES) + (size_t)rank * SHARD_BYTES + ((v >> 2) & 3) * KQ_BYTES + (cch * 4 + (v & 3)) * 4096;
    static_for<8>([&](auto I) { gstore_sys<(I >> 1) * 1024>(r[I], xlane + (I & 1) * H_BYTES, dd); });
    wait_vm<0>();
    flag_store(0u, pv + 1, myws + WS_FLAGS + F_LOC + v * 128);
  } else {
    __builtin_amdgcn_s_sleep(52);
    draw();
  }
  const signed char* wq = p.w + (size_t)(w * 1024) * NFULL + rank * NS;
  {
    unsigned dummy;
    asm volatile("" : "=v"(dummy));
    const signed char* s = wq + (strip & ~3) * 128;
#pragma unroll
    for (int i = 0; i < 2; ++i) {
      const unsigned rr = min((slot - 4) * 19 + 16 * i + (lane >> 2), 511);
      const unsigned o = (((rr >> 6) * 2 + h) * 64 + (rr & 63)) * NFULL + (lane & 3) * 128;
      if (i == 0 || lane < 12) asm volatile("global_load_dword %0, %1, %2" :: "v"(dummy), "v"(o), "s"(s) : "memory");
    }
    const signed char* wsrc0 = wq + strip * 128;
    const unsigned d0 = (lane >> 3) * NFULL + (((lane & 7) ^ ((lane >> 4) & 3)) << 4);
#pragma unroll
    for (int r = 0; r < 2; ++r)
#pragma unroll
      for (int i = 0; i < 8; ++i)
        dma16(d0 ^ ((i & 2) << 5), wsrc0 + (size_t)((8 + r) * 64 + 8 * i) * NFULL, (r == 0 ? lr : lw + 2 * 8192) + i * 1024);
    asm volatile("s_waitcnt vmcnt(16)" :: "v"(dummy) : "memory");
  }
  const unsigned prev = rfl(wait_mov<16>(tick));
  {
    const unsigned call = prev + 1;
    const unsigned base = prev * PROG;
    const unsigned pbase = (prev & 1) * (NR * SHARD_BYTES) + w * KQ_BYTES + h * H_BYTES;
    const u32x4 xrs = make_srd(myws + WS_DATA, X_BYTES);
    const u32x4 frs = make_srd(myws + WS_FLAGS, F_BYTES);
    const u32x4 ors = make_srd(p.out, (unsigned)(NR * MS) * NS * 4u);
    const int lw_ = min(lane, 4 * nloc - 1);
    const unsigned loff = F_LOC + ((lw_ < 8 ? g + 7 * (lw_ >> 2) : 14 + (g >> 1)) * 16 + w * 4 + (lw_ & 3)) * 128;
    const unsigned fa = lane < 4 ? (srcrank * 4 + w) * FSTR + lane * 128 : 0x100000u;
    const unsigned wa = lr + (lane & 15) * 512 + (((lane >> 4) ^ (lane & 7)) << 4);
    const unsigned rrow = w * 4 + (lane >> 5);
    const unsigned ra = lds0 + LDS_RED + rrow * 512 + (((lane & 31) ^ (rrow & 7)) << 4);
    const unsigned o0 = rrow * (NS * 4) + strip * 512 + (lane & 31) * 16;
    unsigned rb = ((lane >> 4) * 16 + ((lane & 15) >> 1)) * 128 + ((((lane & 15) >> 2) | (((lane >> 4) & 1) << 2)) << 4) + (lane & 1) * 8;
    const unsigned d0 = (lane >> 3) * NFULL + (((lane & 7) ^ ((lane >> 4) & 3)) << 4);
    const unsigned oh = h * 16 * (NS * 4);
    const unsigned srcoff = pbase + srcrank * SHARD_BYTES, ocs = srcrank * (MS * NS * 4) + oh - 16 * srcoff;
    const unsigned xbase = (prev & 1) * (NR * SHARD_BYTES) + w * KQ_BYTES + (g & 1) * H_BYTES + rank * SHARD_BYTES;
    asm volatile(R"ASM(
; scalar state
.set sW, 66        ; pair, prologue only
.set sA, 68        ; pair, prologue only
.set sF, 70        ; progress as last seen, at most cap (cap after a timeout)
.set sNF, 71       ; progress the first unit without loads needs
.set sSC, 72       ; scale (NaN after a timeout)
.set sB, 73
.set sCOFF, 74     ; x offset of the current item
.set sNOFF, 75     ; of the next one
.set sLEFT, 76     ; items to go
.set sPROW, 77     ; output offset of the finished item, rows 0-1 of the wave
.set sPROW2, 78    ; rows 2-3
.set sROFF, 79     ; refill offset
.set sCAP, 80
.set sT, 81
.set sT2, 82
.set sN, 83
.set sRET, 84      ; pair
.set sT0, 86       ; pair
.set sT1, 88       ; pair
.set sSMP, 90
.set sFO, 92       ; moves the in-stream flag load out of range while there is nothing to learn from it
.set sG, 98        ; what may be loaded: sF, near the end a little more
.set sEF, 99       ; progress from which on that holds
.set sTENT, 91     ; bit q: slot q was loaded ahead of the progress word and has not been checked
.set sTH, 64       ; 1 while the running unit has to be checked
.set sUO, 65       ; offset of the first unit without loads
.set sTG, 66
.set sSV, 67
.set sRET2, 68     ; pair
.set sPP, 93       ; pace: ticks per item,
.set sPT, 94       ; when the running item is due,
.set sPD, 95       ; naps per unit
.set sOC, 97       ; output offset = 16 x offset + this
; vector registers
.set vWA1, 14
.set vRA1, 15
.set vWL, 10
.set vFL, 11
.set vFB, 12
.set vP, 13
.set vT, 16        ; 16: epilogue temporaries (K steps 0-7), W fragments out of LDS (K steps 8-15)
.set vACC, 32      ; 8 tiles
.set vX, 64        ; 16 fragments
.set vW, 128
; prologue only
.set vRW, 14       ; transposing-read address in slots 0-3
.set vRR, 15       ; in slot 4 (the partial-sum area)
.set vD1, 12
.set vAC2, 96      ; accumulators of item 1

.macro KSOF r               ; round r brings K step ks: 8-15, 4-7, 0-3
  .if (\r) < 8
    .set ks, 8+(\r)
  .elseif (\r) < 12
    .set ks, (\r)-4
  .else
    .set ks, (\r)-12
  .endif
.endm
.macro WDMA1 r, i
  KSOF \r
  .set sl, 2+((\r)+2)%%3
  .if sl == 4
    s_add_u32 m0, %[lr], (\i)*1024
  .else
    s_add_u32 m0, %[lw], sl*8192+(\i)*1024
  .endif
  s_add_u32 s[sA], s[sW], (ks*64+8*(\i))*16384
.endm
.macro WDMA2 i
  s_addc_u32 s[sA+1], s[sW+1], 0
  .if (\i) & 2
    global_load_lds_dwordx4 v[vD1], s[sA:sA+1]
  .else
    global_load_lds_dwordx4 %[d0], s[sA:sA+1]
  .endif
.endm
.macro WREG r, t            ; where tile t of round r goes: tiles 4-7 of K steps 8-15 are parked in a[0:127] until round 12
  KSOF \r
  .if ks < 8
    .set wa, 1
    .set wr, (ks*8+(\t))*4
  .elseif (\t) < 4
    .set wa, 0
    .set wr, vW+((ks-8)*4+(\t))*4
  .else
    .set wa, 1
    .set wr, ((ks-8)*4+(\t)-4)*4
  .endif
.endm
.macro WTR r, t, hf         ; half hf of tile t of round r, transposed out of LDS
  WREG \r, \t
  .set sl, 2+((\r)+2)%%3
  .set va, 11+2*((\t)&1)                        ; v11 / v13 in turn
  .if sl == 4
    .set so, 1024*(\hf)
    .set vb, vRR
  .else
    .set so, sl*8192+1024*(\hf)
    .set vb, vRW
  .endif
  .if (\hf) == 0
    v_xor_b32 v[va], (\t)*16, v[vb]
  .endif
  .if wa
    ds_read_b64_tr_b8 a[wr+2*(\hf):wr+2*(\hf)+1], v[va] offset:so
  .else
    ds_read_b64_tr_b8 v[wr+2*(\hf):wr+2*(\hf)+1], v[va] offset:so
  .endif
.endm
.macro XREG q               ; the fragments of items 0 and 1 for the K step of round q: a ring of six pairs
  .if ((\q)%%6) < 4
    .set xr, vX+8*((\q)%%6)
  .else
    .set xr, vT+8*(((\q)%%6)-4)
  .endif
.endm
.macro XLD q
  XREG \q
  KSOF \q
  s_add_u32 s[sT], s[sCOFF], (ks>>2)*4096
  s_add_u32 s[sT2], s[sNOFF], (ks>>2)*4096
  buffer_load_dwordx4 v[xr:xr+3], %[xlane], %[xrs], s[sT] offen offset:(ks&3)*1024 sc0 sc1
  buffer_load_dwordx4 v[xr+4:xr+7], %[xlane], %[xrs], s[sT2] offen offset:(ks&3)*1024 sc0 sc1
.endm
.macro MFP q, m             ; MFMA m (0-15) of round q: tile m >> 1, item m & 1
  XREG \q
  WREG \q, (\m)>>1
  .set o, vACC+((\m)&1)*(vAC2-vACC)+4*((\m)>>1)
  .set xx, xr+4*((\m)&1)
  .if (\q) == 0 && wa
    v_mfma_i32_16x16x64_i8 v[o:o+3], a[wr:wr+3], v[xx:xx+3], 0
  .elseif (\q) == 0
    v_mfma_i32_16x16x64_i8 v[o:o+3], v[wr:wr+3], v[xx:xx+3], 0
  .elseif wa
    v_mfma_i32_16x16x64_i8 v[o:o+3], a[wr:wr+3], v[xx:xx+3], v[o:o+3]
  .else
    v_mfma_i32_16x16x64_i8 v[o:o+3], v[wr:wr+3], v[xx:xx+3], v[o:o+3]
  .endif
.endm
.macro MFN q, on
  .if \on
    MFP (\q)+(mm>>4), mm&15
    .set mm, mm+1
  .endif
.endm
; round r out of LDS, round r + 2 on its way, nq rounds from q on multiplied, all interleaved
.macro STEP r, q, nq
  .set mm, 0
  .set ii, 0
  .rept 8
    .if (\r) <= 13
      WDMA1 (\r)+2, ii
    .endif
    MFN \q, (\nq)==2
    .if (\r) <= 13
      WDMA2 ii
    .endif
    ; the parked tiles move to their place in LDS: slots 0 and 1 are free all along, slots 2 and 3 after rounds 13 and 14
    .if (\r) >= 14
      .set hn, 8*((\r)-12)+ii
      ds_write_b128 v[vWL], a[4*hn:4*hn+3] offset:hn*1024
    .elseif (\r) >= 4 && (\r) < 12 && (ii&3) == 1
      .set hn, 2*((\r)-4)+(ii>>2)
      ds_write_b128 v[vWL], a[4*hn:4*hn+3] offset:hn*1024
    .endif
    MFN \q, (\nq)>=1
    WTR \r, ii, 0
    MFN \q, (\nq)==2
    WTR \r, ii, 1
    MFN \q, (\nq)>=1
    .set ii, ii+1
  .endr
  s_waitcnt lgkmcnt(0)
.endm
.macro LDF slot, j, soff
  buffer_load_dwordx4 v[vX+16*(\slot)+4*(\j):vX+16*(\slot)+4*(\j)+3], %[xlane], %[xrs], s[\soff] offen offset:(\j)*1024 sc0 sc1
.endm
.macro UOFF dst, qk         ; offset of the unit at position qk, counted from the current item's first
  .if (\qk) < 4
    s_add_u32 s[\dst], s[sCOFF], (\qk)*4096
  .else
    s_add_u32 s[\dst], s[sNOFF], ((\qk)-4)*4096
  .endif
.endm
.macro LDU qk
  UOFF sT, \qk
  LDF (\qk)&3, 0, sT
  LDF (\qk)&3, 1, sT
  LDF (\qk)&3, 2, sT
  LDF (\qk)&3, 3, sT
.endm
.macro ISSUE qk
  LDU \qk
  s_cmp_lt_i32 s[sF], s[sNF]
  s_cselect_b32 s[sT], 1<<((\qk)&3), 0
  s_or_b32 s[sTENT], s[sTENT], s[sT]
  s_add_i32 s[sNF], s[sNF], 1
.endm
.macro GOF
  s_add_i32 s[sG], s[sF], %[eby]
  s_min_i32 s[sG], s[sG], s[sCAP]
  s_cmp_ge_i32 s[sF], s[sEF]
  s_cselect_b32 s[sG], s[sG], s[sF]
.endm
.macro ZCHK q               ; vcc = lanes with a zero dword in slot q
  .set x0, vX+16*(\q)
  v_min3_u32 v[vP], v[x0], v[x0+1], v[x0+2]
  .set t, 3
  .rept 6
    v_min3_u32 v[vP], v[vP], v[x0+t], v[x0+t+1]
    .set t, t+2
  .endr
  v_min_u32 v[vP], v[vP], v[x0+15]
  v_cmp_eq_u32 vcc, 0, v[vP]
  s_nop 3
.endm
.macro REDO q               ; slot q showed a zero dword: load it again behind the progress word (sTG covers it)
L_redo\q\()_%=:
  s_mov_b32 s[sTH], 0
  s_mov_b32 s[sSV], s[sEF]
  s_mov_b32 s[sEF], 0x7fffffff
  s_cmp_ge_i32 s[sF], s[sTG]
  s_cbranch_scc1 1f
  s_call_b64 s[sRET:sRET+1], L_spin%=
1:
  LDU \q
  s_waitcnt vmcnt(0)
  ZCHK \q
  s_cbranch_vccnz 2f                            ; x itself holds zero dwords: no more early loads
  s_mov_b32 s[sEF], s[sSV]
2:
  s_setpc_b64 s[sRET2:sRET2+1]
.endm
.macro CMIN x, n, k         ; step k of the minimum over the n dwords from v[x] on; the last step compares with sTH
  .if (\k) == 0
    v_min3_u32 %[rb], v[\x], v[(\x)+1], v[(\x)+2]
  .elseif 2*(\k)+2 < (\n)
    v_min3_u32 %[rb], %[rb], v[(\x)+2*(\k)+1], v[(\x)+2*(\k)+2]
  .elseif 2*(\k)+2 == (\n)
    v_min_u32 %[rb], %[rb], v[(\x)+2*(\k)+1]
  .else
    v_cmp_gt_u32 vcc, s[sTH], %[rb]
  .endif
.endm
.macro TAKE
  v_max_i32_dpp v[vP], v[vFL], v[vFL] quad_perm:[1,0,3,2] row_mask:0xf bank_mask:0xf
  s_nop 1
  v_max_i32_dpp v[vP], v[vP], v[vP] quad_perm:[2,3,0,1] row_mask:0xf bank_mask:0xf
  s_nop 0
  v_readfirstlane_b32 s[sSMP], v[vP]
  s_nop 0
  s_max_i32 s[sF], s[sF], s[sSMP]
  s_min_i32 s[sF], s[sF], s[sCAP]
  GOF
.endm
; K step f, column tile t: D[W column][x row] += W^T X^T
.macro MFG f, t, acc, x
  .set o, (\acc)+4*(\t)
  .if (\f) == 0
    v_mfma_i32_16x16x64_i8 v[o:o+3], a[4*(\t):4*(\t)+3], v[\x:(\x)+3], 0
  .elseif (\f) < 8
    v_mfma_i32_16x16x64_i8 v[o:o+3], a[((\f)*8+(\t))*4:((\f)*8+(\t))*4+3], v[\x:(\x)+3], v[o:o+3]
  .elseif (\t) < 4
    v_mfma_i32_16x16x64_i8 v[o:o+3], v[vW+(((\f)-8)*4+(\t))*4:vW+(((\f)-8)*4+(\t))*4+3], v[\x:(\x)+3], v[o:o+3]
  .else
    v_mfma_i32_16x16x64_i8 v[o:o+3], v[vT+((\t)-4)*4:vT+((\t)-4)*4+3], v[\x:(\x)+3], v[o:o+3]
  .endif
.endm
.macro MF f, t
  MFG \f, \t, vACC, vX+4*(\f)
.endm
.macro PWG t, acc           ; partial tile t -> LDS
  .if (\t) & 1
    ds_write_b128 v[vWA1], v[(\acc)+4*(\t):(\acc)+4*(\t)+3] offset:((\t)>>1)*128
  .else
    ds_write_b128 %[wa], v[(\acc)+4*(\t):(\acc)+4*(\t)+3] offset:((\t)>>1)*128
  .endif
.endm
.macro PWR t
  PWG \t, vACC
.endm
.macro FIN k, t             ; step k (0-11) of int -> scaled, relu'd float on v[t:t+3]
  .if (\k) < 4
    v_cvt_f32_i32 v[(\t)+(\k)], v[(\t)+(\k)]
  .elseif (\k) < 8
    v_mul_f32 v[(\t)+(\k)-4], s[sSC], v[(\t)+(\k)-4]
  .else
    v_maximum3_f32 v[(\t)+(\k)-8], v[(\t)+(\k)-8], 0, 0     ; NaN-propagating, as np.maximum
  .endif
.endm
.macro OST t, jj
  .set nst, nst+1
  buffer_store_dwordx4 v[\t:(\t)+3], %[o0], %[ors], s[sPROW+(\jj)] offen sc0 sc1
.endm
.macro EPINOW              ; the rest of an epilogue at once
  PWR 3
  PWR 7
  s_waitcnt lgkmcnt(0)
  s_barrier
  .set t, 0
  .rept 4
    ds_read_b128 v[vT+4*t:vT+4*t+3], %[ra] offset:t*8192
    .set t, t+1
  .endr
  .set t, 0
  .rept 4
    ds_read_b128 v[vACC+4*t:vACC+4*t+3], v[vRA1] offset:t*8192
    .set t, t+1
  .endr
  .set jj, 0
  .rept 2
    .set tt, vT+jj*(vACC-vT)
    s_waitcnt lgkmcnt(4-4*jj)
    .set t, 0
    .rept 4
      v_add_u32 v[tt+t], v[tt+t], v[tt+4+t]
      .set t, t+1
    .endr
    .set t, 0
    .rept 4
      v_add3_u32 v[tt+t], v[tt+t], v[tt+8+t], v[tt+12+t]
      .set t, t+1
    .endr
    .set t, 0
    .rept 12
      FIN t, tt
      .set t, t+1
    .endr
    buffer_store_dwordx4 v[tt:tt+3], %[o0], %[ors], s[sPROW+jj] offen sc0 sc1
    .set jj, jj+1
  .endr
.endm
; next operation of the epilogue that rides in the gaps (all tiles but 3 and 7 were written inside the item's last unit)
.macro EOP
  .if e == 0
  .elseif e < 3
    PWR 4*e-1
  .elseif e == 3
    s_waitcnt lgkmcnt(0)
    s_barrier
  .elseif e < 8
    ds_read_b128 v[vT+4*(e-4):vT+4*(e-4)+3], %[ra] offset:(e-4)*8192
  .elseif e == 8 || e == 28 || e == 39
    s_waitcnt lgkmcnt(0)
  .elseif e < 13
    v_add_u32 v[vT+e-9], v[vT+e-9], v[vT+4+e-9]
  .elseif e < 17
    v_add3_u32 v[vT+e-13], v[vT+e-13], v[vT+8+e-13], v[vT+12+e-13]
  .elseif e < 20
    ds_read_b128 v[vT+4*(e-16):vT+4*(e-16)+3], v[vRA1] offset:(e-17)*8192
  .elseif e < 28
    FIN e-20, vT
  .elseif e < 33
    v_add3_u32 v[vT+4+e-29], v[vT+4+e-29], v[vT+8+e-29], v[vT+12+e-29]
  .elseif e == 33
    ds_read_b128 v[vT+8:vT+11], v[vRA1] offset:3*8192
  .elseif e < 38
    FIN e-26, vT
  .elseif e == 38
    OST vT, 0
  .elseif e < 44
    v_add_u32 v[vT+4+e-40], v[vT+4+e-40], v[vT+8+e-40]
  .elseif e < 56
    FIN e-44, vT+4
  .elseif e == 56
    OST vT+4, 1
  .endif
  .set e, e+1
.endm
.macro EGAP g               ; gap g of the item
  .set gi, (\g)&7
  .if e < 3
    EOP
  .elseif e == 3
    .if (\g) == 6
      EOP
    .endif
  .elseif gi == 0 || gi == 7
  .elseif e < 8
    EOP
  .elseif (\g) < 17
  .elseif ((\g) >= 24 && (\g) < 32) || e == 38
    .if gi != 2
      EOP
    .endif
  .else
    EOP
    .if e != 38
      EOP
    .endif
  .endif
.endm
; MFMA i of quarter j of unit q and the gap behind it.  b = how many of the next three units have no loads out:
; b == 0 refills the slot before (offset sROFF) on the way; tail == 1 sends out the first unit without loads.
.macro SLOT q, pend, b, tail
  .set f, 4*(\q)+j
  .if (\q) < 3
    MF f, i
  .elseif i & 1             ; unit 3 finishes tiles j and j + 4 before it turns to the next pair
    .set o, vACC+4*(j+4)
    v_mfma_i32_16x16x64_i8 v[o:o+3], v[vT+2*(i-1):vT+2*(i-1)+3], v[vX+48+2*(i-1):vX+48+2*(i-1)+3], v[o:o+3]
  .else
    MF 12+(i>>1), j
  .endif
  .if i == 0 && (\b) == 0
    LDF ((\q)+3)&3, j, sROFF
  .elseif \tail
    LDF ((\q)+4-(\b))&3, i-4, sUO
  .endif
  .if \pend
    EGAP 8*f+i
  .endif
  .if (\q) == 2 && i == 3
    s_waitcnt lgkmcnt(2)
  .elseif (\q) == 2 && i == 5
    s_waitcnt lgkmcnt(1)
  .endif
  .if f >= 7 && f < 11 && i >= 4
    ds_read_b128 v[vT+4*(i-4):vT+4*(i-4)+3], v[vWL] offset:((f-7)*4+i-4)*1024
  .elseif f == 11 && i >= 4
    ds_read_b128 v[vT+4*(i-4):vT+4*(i-4)+3], v[vWL] offset:i*4*1024
  .endif
  .if (\q) == 3
    .if (i & 1) && j < 3
      ds_read_b128 v[vT+2*(i-1):vT+2*(i-1)+3], v[vWL] offset:((4+(i>>1))*4+j+1)*1024
      .set lc, lc+1
      .if i == 3
        .set p1, lc
      .endif
    .endif
    .if j == 0 && (i & 1) && i < 7 && (\b) == 0
      s_waitcnt vmcnt(12-(i>>1))
    .endif
    .if i == 0
      s_waitcnt lgkmcnt(lc-p1)
    .elseif i == 4
      s_waitcnt lgkmcnt(lc-p3)
    .endif
    .if i == 7
      .set p3, lc
    .endif
    .if j > 0 && (i == 2 || i == 4)
      PWR j-1+2*(i-2)
      .set lc, lc+1
    .endif
  .endif
  .if (\b) == 0
    .if j == 0 && i == 7
      s_add_i32 s[sNF], s[sNF], 1
    .elseif j == 1 && i == 0
      s_add_i32 s[sT2], s[sNF], 10
    .elseif j == 1 && i == 7
      s_cmp_lt_i32 s[sF], s[sT2]
    .elseif j == 2 && i == 0
      s_cselect_b32 s[sFO], 0, 0x100000
    .elseif j == 2 && i == 7
      v_or_b32 v[vFB], s[sFO], %[fa]
    .elseif j == 3 && i == 1
      UOFF sROFF, (\q)+4-4*((\q)/3)
    .elseif j == 3 && i == 2
      v_max_i32_dpp v[vP], v[vFL], v[vFL] quad_perm:[1,0,3,2] row_mask:0xf bank_mask:0xf
      buffer_load_dword v[vFL], v[vFB], %[frs], 0 offen sc0 sc1
    .elseif j == 3 && i == 3
      v_max_i32_dpp v[vP], v[vP], v[vP] quad_perm:[2,3,0,1] row_mask:0xf bank_mask:0xf
    .elseif j == 3 && i == 4
      v_readfirstlane_b32 s[sSMP], v[vP]
    .elseif j == 3 && i == 5
      s_max_i32 s[sF], s[sF], s[sSMP]
    .elseif j == 3 && i == 6
      s_min_i32 s[sF], s[sF], s[sCAP]
    .elseif j == 3 && i == 7
      s_cmp_ge_i32 s[sF], s[sNF]
    .endif
  .else
    .if (\q) < 3 && j == 0
      .if i == 0
        s_waitcnt vmcnt(13-4*(\b))
      .else
        CMIN vX+4*f+4, 12, i-1
      .endif
    .elseif j == 1 && i == 0
      s_mov_b32 s[sPD], 0
    .elseif j == 1 && i == 7
      UOFF sROFF, (\q)+4-4*((\q)/3)
    .elseif j == 2 && i == 0
      UOFF sUO, (\q)+4-(\b)-4*((\q)/3)
    .elseif j == 2 && i == 5
      v_max_i32_dpp v[vP], v[vFL], v[vFL] quad_perm:[1,0,3,2] row_mask:0xf bank_mask:0xf
      buffer_load_dword v[vFL], %[fa], %[frs], 0 offen sc0 sc1
    .elseif j == 2 && i == 6
      v_max_i32_dpp v[vP], v[vP], v[vP] quad_perm:[2,3,0,1] row_mask:0xf bank_mask:0xf
    .elseif j == 2 && i == 7
      v_readfirstlane_b32 s[sSMP], v[vP]
    .elseif j == 3 && i == 0
      s_max_i32 s[sF], s[sF], s[sSMP]
      s_min_i32 s[sF], s[sF], s[sCAP]
    .elseif j == 3 && i == 1
      s_add_i32 s[sG], s[sF], %[eby]
      s_min_i32 s[sG], s[sG], s[sCAP]
    .elseif j == 3 && i == 2
      s_cmp_ge_i32 s[sF], s[sEF]
      s_cselect_b32 s[sG], s[sG], s[sF]
    .elseif j == 3 && i == 3
      s_cmp_ge_i32 s[sG], s[sNF]
    .elseif (\tail) && i == 4
      s_cmp_lt_i32 s[sF], s[sNF]
    .elseif (\tail) && i == 5
      s_cselect_b32 s[sT], 1<<(((\q)+4-(\b))&3), 0
    .elseif (\tail) && i == 6
      s_or_b32 s[sTENT], s[sTENT], s[sT]
      s_add_i32 s[sNF], s[sNF], 1
    .elseif (\tail) && i == 7
      .if (\b) == 3
        buffer_load_dword v[vFL], %[fa], %[frs], 0 offen sc0 sc1
      .endif
      s_cmp_ge_i32 s[sG], s[sNF]
    .endif
  .endif
  .if (\b) == 0 && (\pend) && (\q) == 0
    .if j == 0 && i == 3
      s_memrealtime s[sT1:sT1+1]                ; back behind the lgkmcnt(0) of gap 6
    .elseif j == 1 && i == 5
      s_sub_i32 s[sPD], s[sPT], s[sT1]
    .elseif j == 1 && i == 6
      s_ashr_i32 s[sPD], s[sPD], 4
    .endif
  .endif
  .if (\b) == 0 && j == 2 && i == 1
    s_cmp_gt_i32 s[sPD], 0
    s_cbranch_scc1 L_z\q\()_\pend\()_%=
L_y\q\()_\pend\()_%=:
  .endif
  ; on to the next item
  .if f == 9 && i == 1
    s_lshl_b32 s[sPROW], s[sCOFF], 4
    s_add_u32 s[sPROW], s[sPROW], s[sOC]
  .elseif f == 9 && i == 2
    s_add_u32 s[sPROW2], s[sPROW], 2*8192
    s_mov_b32 s[sOC], %[ocs]
  .elseif f == 12 && i == 2
    s_mov_b32 s[sCOFF], s[sNOFF]
    s_add_u32 s[sNOFF], s[sNOFF], 16384
  .elseif f == 12 && i == 4
    s_barrier                                   ; every wave has read the previous partials
  .elseif f == 12 && i == 6
    s_sub_i32 s[sLEFT], s[sLEFT], 1
    s_add_i32 s[sPT], s[sPT], s[sPP]
  .endif
  .set i, i+1
.endm
; Unit q.  b == 0 ends with SCC = "the next refill is available".  b > 0 checks a unit that was loaded early, and in its
; last four gaps either sends out another unit (then SCC = "more is available") or leaves for TAILB.
.macro SECT q, p, b
L_s\b\()_\q\()_\p\()_%=:
  .if (\q) == 0
    .set e, 0
  .elseif (\q) == 1
    .set e, 25
  .else
    .set e, 57
  .endif
  .set nst, 0               ; output stores issued inside this unit so far
  .set lc, 4                ; unit 3: LDS operations issued so far, and where the reads into temporaries 1 and 3 stand
  .set p1, 2
  .set p3, 4
  .if (\b) > 0
    s_bfe_u32 s[sTH], s[sTENT], 0x10000+(\q)
    s_bitset0_b32 s[sTENT], \q
    .if (\q) < 3
      .set nd, 4
      s_waitcnt vmcnt(16-4*(\b))
    .else
      .set nd, 16
      s_waitcnt vmcnt(13-4*(\b))
      s_cbranch_scc0 L_r0\b\()_\q\()_\p\()_%=
    .endif
    .set t, 0
    .rept nd/2+1
      CMIN vX+16*(\q), nd, t
      .set t, t+1
    .endr
    s_cbranch_vccnz L_x0\b\()_\q\()_\p\()_%=
L_r0\b\()_\q\()_\p\()_%=:
  .endif
  .set j, 0
  .rept 4
    .if (\b) == 0 && ((\q) < 3 || j == 0)
      s_waitcnt vmcnt(12+nst)
    .elseif (\b) > 0 && (\q) < 3 && j == 1
      s_cbranch_vccnz L_x1\b\()_\q\()_\p\()_%=
L_r1\b\()_\q\()_\p\()_%=:
    .endif
    .set i, 0
    .rept 4
      SLOT \q, \p, \b, 0
    .endr
    .if j == 3 && (\b) > 0 && !((\q) == 3 && (\b) == 3)
      s_cbranch_scc0 L_t\b\()_\q\()_\p\()_%=
      .rept 4
        SLOT \q, \p, \b, 1
      .endr
    .else
      .rept 4
        SLOT \q, \p, \b, 0
      .endr
    .endif
    .set j, j+1
  .endr
.endm
.macro TAILB q, p, b, to    ; out of line: the end of unit q when nothing can be sent, and the ways out to REDO
  .if !((\q) == 3 && (\b) == 3)
L_t\b\()_\q\()_\p\()_%=:
    .if (\q) == 0
      .set e, 22
    .else
      .set e, 57
    .endif
    .set lc, 1
    .set p3, 0
    .set j, 3
    .set i, 4
    .rept 4
      SLOT \q, \p, \b, 0
    .endr
    s_branch \to
  .endif
L_x0\b\()_\q\()_\p\()_%=:
  s_sub_i32 s[sTG], s[sNF], 4-(\b)
  s_call_b64 s[sRET2:sRET2+1], L_redo\q\()_%=
  s_branch L_r0\b\()_\q\()_\p\()_%=
  .if (\q) < 3
L_x1\b\()_\q\()_\p\()_%=:
  s_sub_i32 s[sTG], s[sNF], 4-(\b)
  s_call_b64 s[sRET2:sRET2+1], L_redo\q\()_%=
  s_branch L_r1\b\()_\q\()_\p\()_%=
  .endif
.endm
.macro PAUSE q, p
L_z\q\()_\p\()_%=:
  s_min_i32 s[sT], s[sPD], 6
1:
  s_sleep 1
  s_sub_i32 s[sT], s[sT], 1
  s_cmp_gt_i32 s[sT], 0
  s_cbranch_scc1 1b
  s_branch L_y\q\()_\p\()_%=
.endm
; in front of unit q: entry c<n> = n of the units q, q + 1, q + 2 have no loads out yet
.macro CHAIN q, p
L_c3_\q\()_\p\()_%=:
  s_cmp_ge_i32 s[sG], s[sNF]
  s_cbranch_scc1 1f
  s_mov_b32 s[sTG], s[sNF]
  s_call_b64 s[sRET:sRET+1], L_spin%=
1:
  ISSUE \q
  buffer_load_dword v[vFL], %[fa], %[frs], 0 offen sc0 sc1       ; a look or an output store follows every unit's loads
L_c2_\q\()_\p\()_%=:
  s_cmp_ge_i32 s[sG], s[sNF]
  s_cbranch_scc0 L_s3_\q\()_\p\()_%=
  ISSUE (\q)+1
L_c1_\q\()_\p\()_%=:
  s_cmp_ge_i32 s[sG], s[sNF]
  s_cbranch_scc0 L_s2_\q\()_\p\()_%=
  ISSUE (\q)+2
L_c0_\q\()_\p\()_%=:
  s_cmp_eq_u32 s[sTENT], 0                      ; the refilling variants check nothing
  s_cselect_b32 s[sT], s[sNF], 0x7fffffff
  s_cmp_ge_i32 s[sF], s[sT]
  s_cbranch_scc1 L_s0_\q\()_\p\()_%=
  s_branch L_s1_\q\()_\p\()_%=
.endm
.macro SIDE b, c, n         ; the variants with b units behind; c = b - 1, n = where a unit goes when nothing could be sent
  SECT 0, 0, \b
  s_cbranch_scc1 L_c\c\()_1_0_%=
  SECT 1, 0, \b
  s_cbranch_scc1 L_c\c\()_2_0_%=
  s_branch L_s\b\()_2_0_%=
  SECT 0, 1, \b
  s_cbranch_scc1 L_c\c\()_1_1_%=
  SECT 1, 1, \b
  s_cbranch_scc1 L_c\c\()_2_0_%=
  SECT 2, 0, \b
  s_cbranch_scc1 L_c\c\()_3_0_%=
  SECT 3, 0, \b
  .if (\b) == 3
    s_branch L_fin%=
  .else
    s_cbranch_scc1 L_c\c\()_0_1_%=
    s_branch L_s\b\()_0_1_%=
  .endif
  TAILB 0, 0, \b, L_\n\()_1_0_%=
  TAILB 1, 0, \b, L_\n\()_2_0_%=
  TAILB 0, 1, \b, L_\n\()_1_1_%=
  TAILB 1, 1, \b, L_\n\()_2_0_%=
  TAILB 2, 0, \b, L_\n\()_3_0_%=
  TAILB 3, 0, \b, L_\n\()_0_1_%=
.endm

  s_nop 4
  buffer_load_dword v[vT], %[loff], %[frs], 0 offen sc0 sc1      ; the re-tiling flags
  s_mov_b64 s[sW:sW+1], %[wsrc]
  v_xor_b32 v[vD1], 64, %[d0]
  v_add_u32 v[vRW], %[lw], %[rb]
  v_add_u32 v[vRR], %[lr], %[rb]
  v_add_u32 v[vWL], %[lw], %[xlane]
  s_add_i32 s[sCAP], %[base], 64
  s_add_i32 s[sEF], %[base], %[efrom]
  s_mov_b32 s[sTENT], 0
  s_mov_b32 s[sF], %[base]
  s_lshl_b32 s[sT], %[nloc], 2
  s_sub_i32 s[sNF], %[base], s[sT]
  s_add_i32 s[sNF], s[sNF], 9                   ; the stream starts with item 2
  s_mov_b32 s[sSC], %[scale]
  s_mov_b32 s[sCOFF], %[coff]
  s_add_u32 s[sNOFF], %[coff], 7*16384
  ; W round by round; items 0 and 1 (own shard) are multiplied on the way.  Each vmcnt counts the operations issued
  ; behind the one that is needed (8 per round, 2 per K step of x).
  s_waitcnt vmcnt(9)
  STEP 0, 0, 0
  s_waitcnt vmcnt(8)                            ; round 1 and the look at the flags
  v_subrev_u32 v[vT], %[call], v[vT]
  v_cmp_le_i32 vcc, 0, v[vT]
  s_nop 1
  s_cmp_eq_u64 vcc, exec
  s_cbranch_scc1 L_lgo%=
  s_memrealtime s[sT0:sT0+1]
L_lwait%=:
  v_mov_b32 v[vT+1], %[ftmo]
  buffer_load_dword v[vT], %[loff], %[frs], 0 offen sc0 sc1
  buffer_load_dword v[vT+1], v[vT+1], %[frs], 0 offen sc0 sc1
  s_memrealtime s[sT1:sT1+1]
  s_waitcnt vmcnt(0) lgkmcnt(0)
  v_subrev_u32 v[vT], %[call], v[vT]
  v_cmp_le_i32 vcc, 0, v[vT]
  v_readfirstlane_b32 s[sT], v[vT+1]
  s_nop 1
  s_cmp_eq_u64 vcc, exec
  s_cbranch_scc1 L_lgo%=
  s_sub_u32 s[sT1], s[sT1], s[sT0]
  s_cmp_eq_u32 s[sT], %[call]
  s_cselect_b32 s[sT1], -1, s[sT1]
  s_cmp_lt_u32 s[sT1], %[dl]
  s_cbranch_scc1 L_lwait%=
  s_call_b64 s[sRET:sRET+1], L_giveup%=
L_lgo%=:
  XLD 0
  XLD 1
  XLD 2
  XLD 3
  STEP 1, 0, 0
  s_waitcnt vmcnt(16)
  XLD 4
  XLD 5
  STEP 2, 0, 0
  s_waitcnt vmcnt(12)                           ; round 3 and the x loaded in front of it
  STEP 3, 0, 2
  s_waitcnt vmcnt(8)
  XLD 6
  XLD 7
  STEP 4, 2, 2
  s_waitcnt vmcnt(12)
  XLD 8
  XLD 9
  STEP 5, 4, 1
  s_waitcnt vmcnt(12)
  XLD 10
  STEP 6, 5, 1
  s_waitcnt vmcnt(10)
  XLD 11
  STEP 7, 6, 1
  s_waitcnt vmcnt(10)
  XLD 12
  STEP 8, 7, 1
  s_waitcnt vmcnt(10)
  XLD 13
  STEP 9, 8, 1
  s_waitcnt vmcnt(10)
  XLD 14
  STEP 10, 9, 1
  s_waitcnt vmcnt(10)
  XLD 15
  STEP 11, 10, 1
  s_waitcnt vmcnt(10)
  buffer_load_dword v[vT], %[fa], %[frs], 0 offen sc0 sc1        ; a first look at the source's progress
  STEP 12, 11, 1
  s_lshl_b32 s[sPROW], s[sCOFF], 4
  s_add_u32 s[sPROW], s[sPROW], %[ocr]
  s_add_u32 s[sPROW2], s[sPROW], 2*8192
  s_mov_b32 s[sCOFF], %[c2off]
  s_mov_b32 s[sNOFF], %[c3off]
  s_mov_b32 s[sOC], %[oc2]
  s_add_i32 s[sLEFT], %[nloc], 14
  s_add_u32 s[sROFF], s[sCOFF], 3*4096
  s_waitcnt vmcnt(9)
  STEP 13, 12, 1
  s_waitcnt vmcnt(8)                            ; round 14 and the look
  v_mov_b32 v[vFL], v[vT]
  s_nop 1
  TAKE
  STEP 14, 13, 1
  s_waitcnt vmcnt(0)
  s_mov_b32 s[sB], 3
  s_cmp_ge_i32 s[sF], s[sNF]
  s_cbranch_scc0 L_p15%=
  ISSUE 0
  s_mov_b32 s[sB], 2
L_p15%=:
  STEP 15, 14, 1
  v_mov_b32 v[vFL], 0
  v_xor_b32 v[vWA1], 64, %[wa]
  v_xor_b32 v[vRA1], 32, %[ra]
  v_add_u32 v[vRA1], 1024, v[vRA1]
  v_mov_b32 v[vFB], %[fa]
  .set mm, 0
  .rept 8
    MFP 15, 2*mm
    .set mm, mm+1
  .endr
  .set mm, 0
  .rept 8
    MFP 15, 2*mm+1
    .if (mm & 3) != 3
      PWR mm
    .endif
    .set mm, mm+1
  .endr
  s_cmp_ge_i32 s[sF], s[sNF]
  s_cbranch_scc0 L_pe%=
  ISSUE 1
  s_mov_b32 s[sB], 1
L_pe%=:
  .set nst, 0
  EPINOW
  s_barrier                                     ; every wave has read the partials of item 0
  PWG 0, vAC2
  PWG 1, vAC2
  PWG 2, vAC2
  PWG 4, vAC2
  PWG 5, vAC2
  PWG 6, vAC2
  .set t, 12
  .rept 4
    v_mov_b32 v[vACC+t], v[vAC2+t]
    v_mov_b32 v[vACC+16+t], v[vAC2+16+t]
    .set t, t+1
  .endr
  s_memrealtime s[sT1:sT1+1]
  s_cmp_eq_u32 %[nloc], 3
  s_movk_i32 s[sT], 256
  s_cselect_b32 s[sT], 241, s[sT]               ; 4096 / items
  s_waitcnt lgkmcnt(0)
  s_mov_b32 s[sPT], s[sT1]
  s_sub_i32 s[sPP], %[tgoal], s[sT1]
  s_mul_i32 s[sPP], s[sPP], s[sT]
  s_ashr_i32 s[sPP], s[sPP], 12
  s_mov_b32 s[sPD], 0
  s_add_u32 s[sPROW], s[sPROW], 7*16384*16
  s_add_u32 s[sPROW2], s[sPROW2], 7*16384*16
  s_cmp_eq_u32 s[sB], 1
  s_cbranch_scc1 L_c1_0_1_%=
  s_cmp_eq_u32 s[sB], 2
  s_cbranch_scc1 L_c2_0_1_%=
  s_branch L_c3_0_1_%=

  SECT 0, 0, 0
  s_cbranch_scc0 L_s1_1_0_%=
  SECT 1, 0, 0
  s_cbranch_scc0 L_s1_2_0_%=
  s_branch L_s0_2_0_%=
  SECT 0, 1, 0
  s_cbranch_scc0 L_s1_1_1_%=
  SECT 1, 1, 0
  s_cbranch_scc0 L_s1_2_0_%=
  SECT 2, 0, 0
  s_cbranch_scc0 L_s1_3_0_%=
  SECT 3, 0, 0
  s_cbranch_scc1 L_s0_0_1_%=
  s_branch L_s1_0_1_%=
  PAUSE 0, 0
  PAUSE 1, 0
  PAUSE 0, 1
  PAUSE 1, 1
  PAUSE 2, 0
  PAUSE 3, 0
  SIDE 1, 0, s2
  SIDE 2, 1, s3
  SIDE 3, 2, c3
  CHAIN 0, 0
  CHAIN 1, 0
  CHAIN 0, 1
  CHAIN 1, 1
  CHAIN 2, 0
  CHAIN 3, 0

L_fin%=:                                        ; nothing of the next item is in flight: finish this one now
  s_mov_b32 s[sB], 3
  s_cmp_ge_i32 s[sG], s[sNF]
  s_cbranch_scc0 L_epi%=
  ISSUE 0
  s_mov_b32 s[sB], 2
  s_cmp_ge_i32 s[sG], s[sNF]
  s_cbranch_scc0 L_epi%=
  ISSUE 1
  s_mov_b32 s[sB], 1
  s_cmp_ge_i32 s[sG], s[sNF]
  s_cbranch_scc0 L_epi%=
  ISSUE 2
  s_mov_b32 s[sB], 0
L_epi%=:
  s_nop 7                                       ; the last MFMAs -> ds_write of their results
  EPINOW
  s_cmp_eq_u32 s[sLEFT], 0
  s_cbranch_scc1 L_done%=
  TAKE
  s_cmp_eq_u32 s[sB], 3
  s_cbranch_scc1 L_c3_0_0_%=
  s_cmp_eq_u32 s[sB], 2
  s_cbranch_scc1 L_c2_0_0_%=
  s_cmp_eq_u32 s[sB], 1
  s_cbranch_scc1 L_c1_0_0_%=
  s_branch L_c0_0_0_%=

  REDO 0
  REDO 1
  REDO 2
  REDO 3
L_spin%=:                                       ; until sG >= sTG
  s_memrealtime s[sT0:sT0+1]
  s_mov_b32 s[sN], 0
L_spin1%=:
  buffer_load_dword v[vFL], %[fa], %[frs], 0 offen sc0 sc1
  s_waitcnt vmcnt(0)
  TAKE
  s_cmp_ge_i32 s[sG], s[sTG]
  s_cbranch_scc1 L_back%=
  s_sleep 1
  s_add_i32 s[sN], s[sN], 1
  s_and_b32 s[sT], s[sN], 15
  s_cbranch_scc1 L_spin1%=
  ; now and then the timeout word and the clock
  v_mov_b32 v[vP], %[ftmo]
  buffer_load_dword v[vP], v[vP], %[frs], 0 offen sc0 sc1
  s_memrealtime s[sT1:sT1+1]
  s_waitcnt vmcnt(0) lgkmcnt(0)
  v_readfirstlane_b32 s[sT], v[vP]
  s_sub_u32 s[sT1], s[sT1], s[sT0]
  s_nop 0
  s_cmp_eq_u32 s[sT], %[call]
  s_cselect_b32 s[sT1], -1, s[sT1]
  s_cmp_lt_u32 s[sT1], %[dl]
  s_cbranch_scc1 L_spin1%=
L_giveup%=:                                     ; publish the timeout, wait no more, store NaN from now on
  v_mov_b32 v[vP], %[ftmo]
  v_mov_b32 v[vFL], %[call]
  s_mov_b64 exec, 1
  buffer_store_dword v[vFL], v[vP], %[frs], 0 offen sc1
  s_mov_b64 exec, -1
  s_mov_b32 s[sF], s[sCAP]
  s_mov_b32 s[sG], s[sCAP]
  s_mov_b32 s[sSC], 0x7fc00000
L_back%=:
  s_setpc_b64 s[sRET:sRET+1]
L_done%=:
gemm_a2a_code_end:
  s_endpgm
.purgem KSOF
.purgem WREG
.purgem XREG
.purgem XLD
.purgem MFP
.purgem STEP
.purgem MFG
.purgem PWG
.purgem EPINOW
.purgem WDMA1
.purgem WDMA2
.purgem MFN
.purgem WTR
.purgem LDF
.purgem UOFF
.purgem ISSUE
.purgem LDU
.purgem GOF
.purgem ZCHK
.purgem REDO
.purgem CMIN
.purgem SLOT
.purgem TAILB
.purgem TAKE
.purgem MF
.purgem PWR
.purgem FIN
.purgem OST
.purgem EOP
.purgem EGAP
.purgem SECT
.purgem CHAIN
.purgem PAUSE
.purgem SIDE
)ASM"
        : [rb] "+&v"(rb)
        : [xrs] "s"(xrs), [frs] "s"(frs), [ors] "s"(ors), [wsrc] "s"(wq + strip * 128), [lw] "s"(lw), [lr] "s"(lr),
          [base] "s"(base), [call] "s"(call), [coff] "s"(pbase + rank * SHARD_BYTES + g * 16384),
          [c2off] "s"(extra ? xbase + (14 + (g >> 1)) * 16384 : srcoff), [c3off] "s"(extra ? srcoff : srcoff + 16384),
          [nloc] "s"(nloc), [ocr] "s"(rank * (MS * NS * 4) + oh - 16 * (pbase + rank * SHARD_BYTES)), [ocs] "s"(ocs),
          [oc2] "s"(extra ? rank * (MS * NS * 4) + (g & 1) * 16 * (NS * 4) - 16 * xbase : ocs),
          [scale] "s"(rfl(__builtin_bit_cast(unsigned, scale))), [tgoal] "s"(tgoal),
          [xlane] "v"(xlane), [wa] "v"(wa), [ra] "v"(ra), [o0] "v"(o0), [fa] "v"(fa), [d0] "v"(d0), [loff] "v"(loff),
          [eby] "i"(EARLY_BY), [efrom] "i"(EARLY_FROM), [dl] "i"((unsigned)DEADLINE_TICKS), [ftmo] "i"(F_TMO)
        : "memory", "vcc", "scc", ACLOB, VCLOB, SCLOB);
  }
}

extern "C" void kernel_launch(int rank, int nranks, ncclComm_t comm,
                              hipStream_t stream,
                              void* const* d_in, const int* in_sizes, int n_in,
                              void* d_out, int out_size,
                              const int* in_dtypes, int out_dtype,
                              void* const* d_out_all,
                              void* const* d_ws_all, size_t ws_bytes) {
  (void)comm; (void)in_dtypes; (void)out_dtype; (void)d_out_all;
  if (nranks != NR || n_in != 4 || (unsigned)rank >= (unsigned)NR || ws_bytes < WS_NEED ||
      in_sizes[0] != MS * KD || in_sizes[1] != KD * NFULL || out_size != NR * MS * NS) {
    fprintf(stderr, "kernel_launch: unexpected problem shape\n");
    return;
  }
  Params p;
  p.x = (const signed char*)d_in[0];
  p.w = (const signed char*)d_in[1];
  p.sx = (const float*)d_in[2];
  p.sw = (const float*)d_in[3];
  p.out = (float*)d_out;
  p.myws = (char*)d_ws_all[rank];
  p.pad = nullptr;
  p.rank = rank;
  {
    const unsigned set1 = 0x4D;
    unsigned tab = 0;
    int na = 0, nb = 4;
    for (int dr = 1; dr < NR; ++dr) {
      const int d = (rank + dr) % NR;
      const bool same = ((set1 >> d) & 1) == ((set1 >> rank) & 1);
      tab |= (unsigned)dr << (4 * (same ? na++ : nb++));
    }
    for (int i = 0; i < NR; ++i) p.wsx[i] = (char*)d_ws_all[(rank + ((tab >> (4 * i)) & 7)) % NR];
    p.xcc_tab = tab;
  }
  hipLaunchKernelGGL(gemm_a2a_kernel, dim3(NWG), dim3(NTHR), 0, stream, p);
  HIP_CHECK(hipGetLastError());
}
```
